```python
import numpy as np

kernel_name = "gpu_gemm_ar_m4096_k4096_n16384_w8a8_dequant"


ABSMAX_RTOL = 0.001

COMPUTE_DTYPE = "int8"
EPILOGUE = "dequant"
IN_QUANT = "int8"
OUT_QUANT = None
IN_DTYPES = {"x": "int8", "w_mat": "int8",
             "scale_x": "float32", "scale_w": "float32"}
WORLD_SIZE = 8

def setup_inputs():

    x = np.random.randint(-127, 128, size=(4096, 4096)).astype(np.int8)
    w_mat = np.random.randint(-127, 128, size=(4096, 16384)).astype(np.int8)
    scale_x = (0.02 * (1.0 + np.random.rand(1))).astype(np.float32)
    scale_w = (0.02 * (1.0 + np.random.rand(1))).astype(np.float32)
    return {"x": x, "w_mat": w_mat, "scale_x": scale_x, "scale_w": scale_w}

def reference(x, w_mat, scale_x, scale_w):
    acc = x.astype(np.float64) @ w_mat.astype(np.float64)
    y = (acc.astype(np.float32) * (scale_x * scale_w)).astype(np.float32)
    return y
```

```cpp
#include <hip/hip_runtime.h>
#include <rccl/rccl.h>
#include <cstdio>
#include <cstdint>

#define HIP_CHECK(expr) do { hipError_t _e = (expr); if (_e != hipSuccess) { \
    fprintf(stderr, "HIP error %s at %s:%d\n", hipGetErrorString(_e), __FILE__, __LINE__); \
    return; } } while (0)

namespace {

#define LAS __attribute__((address_space(3)))
typedef int i32x4 __attribute__((ext_vector_type(4)));
typedef float f32x4 __attribute__((ext_vector_type(4)));

constexpr int M = 4096, N = 16384, K = 4096, NR = 8;
constexpr int KS = K / NR;
constexpr int IMG = 16384;
constexpr int PIECE = 8 * IMG;
constexpr int PANEL = NR * PIECE;
constexpr int NPX = M / 256, NPW = N / 256, NP = NPX + NPW;
constexpr int NQ = K / 128;
constexpr int KSTEP = 2 * IMG;
constexpr int STAGE_BYTES = 8 * IMG;
constexpr int LDS_BYTES = STAGE_BYTES + 256;

constexpr size_t OFF_TICKET = 0, OFF_TMO = 128, OFF_CNT = 4096, OFF_STG = 1u << 20;
constexpr size_t WS_NEED = OFF_STG + (size_t)2 * NP * PANEL;
constexpr unsigned long long DEADLINE_TICKS = 20000000ull;

constexpr int GRID = 256;
constexpr int NC = 16;
constexpr int NCX = NC / 8;
constexpr int NGX = GRID / 8 - NCX;
constexpr int NW = NC * 8;
constexpr int ITEMS = (128 * (NR - 1) + NW - 1) / NW;

struct Params {
    const signed char* x; const signed char* w; const float* sx; const float* sw; float* out;
    void* const* ws_all; int rank;
};

__device__ __forceinline__ unsigned* cnt_ptr(unsigned char* ws, int panel, int src) {
    return (unsigned*)(ws + OFF_CNT + (size_t)(panel * NR + src) * 128);
}
__device__ __forceinline__ int panel_of_seq(int s) {
    if (s < 32) { const int i = s >> 1; if (s & 1) return NPX + i; return (i < 8) ? 2 * i : 2 * (i - 8) + 1; }
    return NPX + 16 + (s - 32);
}
__device__ __forceinline__ void st_wt_sys(void* p, i32x4 v) {
    asm volatile("global_store_dwordx4 %0, %1, off sc0 sc1\n\ts_nop 1" :: "v"(p), "v"(v) : "memory");
}
__device__ __forceinline__ unsigned ld_relaxed_sys(const unsigned* p) { return __hip_atomic_load(p, __ATOMIC_RELAXED, __HIP_MEMORY_SCOPE_SYSTEM); }

struct Unit { int pm, pn; };
__device__ __forceinline__ Unit unit_of(int j, int e) {
    const int h = 8 + j; Unit u;
    if (e < h) { u.pm = 2 * j; u.pn = e; }
    else if (e < 2 * h) { u.pm = 2 * j + 1; u.pn = e - h; }
    else { const int f = e - 2 * h; u.pn = h + (f >> 1); u.pm = 2 * j + (f & 1); }
    return u;
}

}

__global__ __launch_bounds__(512, 2) void fused_gather_gemm(Params P) {
    extern __shared__ __attribute__((aligned(16))) unsigned char shm[];
    LAS unsigned char* lds = (LAS unsigned char*)shm;
    LAS unsigned* ctl = (LAS unsigned*)(lds + STAGE_BYTES);
    const int tid = threadIdx.x, wid = __builtin_amdgcn_readfirstlane(tid >> 6), lane = tid & 63;
    const int rank = P.rank;
    unsigned char* ws = (unsigned char*)P.ws_all[rank];
    unsigned* tmo = (unsigned*)(ws + OFF_TMO);
    if (tid == 0) {
        const unsigned t = __hip_atomic_fetch_add((unsigned*)(ws + OFF_TICKET), 1u, __ATOMIC_RELAXED, __HIP_MEMORY_SCOPE_AGENT);
        ctl[0] = t / gridDim.x + 1u; ctl[1] = 0u;
    }
    __syncthreads();
    const unsigned call = __builtin_amdgcn_readfirstlane(ctl[0]);
    unsigned char* stg = ws + OFF_STG + (size_t)(call & 1u) * NP * PANEL;

    {
        const int img = wid * GRID + (int)blockIdx.x;
        if (img < NP * 8) {
            const int panel = panel_of_seq(img >> 3), sub = img & 7, kt = sub >> 1, half = sub & 1;
            unsigned char* dst = stg + (size_t)panel * PANEL + (size_t)rank * PIECE + (size_t)sub * IMG;
            if (panel < NPX) {
                const signed char* src = P.x + (size_t)(panel * 256 + half * 128 + (lane & 15)) * KS + kt * 128 + (lane >> 4) * 16;
                i32x4 v[16];
#pragma unroll
                for (int i = 0; i < 16; ++i) v[i] = *(const i32x4*)(src + (size_t)(i >> 1) * 16 * KS + (i & 1) * 64);
#pragma unroll
                for (int i = 0; i < 16; ++i) st_wt_sys(dst + i * 1024 + lane * 16, v[i]);
            } else {
                const int g = lane >> 3, c8 = lane & 7;
                const signed char* src = P.w + (size_t)(kt * 128 + 16 * g) * N + (panel - NPX) * 256 + half * 128 + c8 * 16;
                i32x4 v[16];
#pragma unroll
                for (int j = 0; j < 16; ++j) v[j] = *(const i32x4*)(src + (size_t)j * N);
                LAS unsigned char* tb = lds + wid * IMG;
                i32x4 o[16];
#pragma unroll
                for (int e = 0; e < 4; ++e)
#pragma unroll
                    for (int d = 0; d < 4; ++d) {
                        const unsigned r0 = v[4 * e][d], r1 = v[4 * e + 1][d], r2 = v[4 * e + 2][d], r3 = v[4 * e + 3][d];
                        const unsigned t0 = __builtin_amdgcn_perm(r1, r0, 0x05010400u), t1 = __builtin_amdgcn_perm(r1, r0, 0x07030602u);
                        const unsigned t2 = __builtin_amdgcn_perm(r3, r2, 0x05010400u), t3 = __builtin_amdgcn_perm(r3, r2, 0x07030602u);
                        o[4 * d + 0][e] = (int)__builtin_amdgcn_perm(t2, t0, 0x05040100u);
                        o[4 * d + 1][e] = (int)__builtin_amdgcn_perm(t2, t0, 0x07060302u);
                        o[4 * d + 2][e] = (int)__builtin_amdgcn_perm(t3, t1, 0x05040100u);
                        o[4 * d + 3][e] = (int)__builtin_amdgcn_perm(t3, t1, 0x07060302u);
                    }
#pragma unroll
                for (int c = 0; c < 16; ++c) *(LAS i32x4*)(tb + c8 * 2048 + g * 256 + ((c ^ c8) * 16)) = o[c];
                asm volatile("s_waitcnt lgkmcnt(0)" ::: "memory");
                __builtin_amdgcn_wave_barrier();
#pragma unroll
                for (int i = 0; i < 16; ++i) {
                    const int rc8 = i >> 1, rg = (i & 1) * 4 + (lane >> 4), rc = lane & 15;
                    const i32x4 t = *(const LAS i32x4*)(tb + rc8 * 2048 + rg * 256 + ((rc ^ rc8) * 16));
                    st_wt_sys(dst + i * 1024 + lane * 16, t);
                }
            }
            asm volatile("s_waitcnt vmcnt(0)" ::: "memory");
            if (lane == 0) __hip_atomic_fetch_add(cnt_ptr(ws, panel, rank), 1u, __ATOMIC_RELAXED, __HIP_MEMORY_SCOPE_SYSTEM);
        }
        __syncthreads();
    }

    const int xg = (int)blockIdx.x & 7, xi = (int)blockIdx.x >> 3;
    int e0, estep, eend;
    if (xi < NCX) {
        const int cw = (int)blockIdx.x * 8 + wid;
        unsigned char* dstb[ITEMS]; int boff[ITEMS]; bool val[ITEMS];
#pragma unroll
        for (int it = 0; it < ITEMS; ++it) {
            const int id = cw + it * NW; val[it] = id < 128 * (NR - 1);
            const int b = id & 127, d = ((id >> 7) + b) % (NR - 1);
            const unsigned peer = (unsigned)(rank + 1 + d) % (unsigned)NR;
            boff[it] = b * 1024 + lane * 16;
            dstb[it] = (unsigned char*)P.ws_all[peer < NR ? peer : 0] + OFF_STG + (size_t)(call & 1u) * NP * PANEL + (size_t)rank * PIECE + boff[it];
        }
        unsigned* rcnt = nullptr;
        if (wid == 0 && lane < NR - 1) rcnt = (unsigned*)((unsigned char*)P.ws_all[(unsigned)(rank + 1 + lane) % (unsigned)NR] + OFF_CNT + (size_t)rank * 128);
        const unsigned char* srcb = stg + (size_t)rank * PIECE;
        bool dead = false;
        auto wait_local = [&](int panel) {
            const unsigned* c = cnt_ptr(ws, panel, rank); const unsigned need = 8u * call;
            if (ld_relaxed_sys(c) < need) {
                const unsigned long long t0 = __builtin_amdgcn_s_memrealtime();
                while (ld_relaxed_sys(c) < need) {
                    if (__hip_atomic_load(tmo, __ATOMIC_RELAXED, __HIP_MEMORY_SCOPE_AGENT) == call || __builtin_amdgcn_s_memrealtime() - t0 > DEADLINE_TICKS) {
                        __hip_atomic_store(tmo, call, __ATOMIC_RELAXED, __HIP_MEMORY_SCOPE_AGENT); dead = true; break; }
                    __builtin_amdgcn_s_sleep(2);
                }
            }
            __builtin_amdgcn_fence(__ATOMIC_ACQUIRE, "agent");
            asm volatile("s_waitcnt vmcnt(0)" ::: "memory");
        };
        i32x4 v[ITEMS];
        for (int s = 0; s < NP; ++s) {
            const int panel = panel_of_seq(s);
            wait_local(panel);
            const size_t po = (size_t)panel * PANEL;
#pragma unroll
            for (int it = 0; it < ITEMS; ++it) if (val[it]) v[it] = *(const i32x4*)(srcb + po + boff[it]);
#pragma unroll
            for (int it = 0; it < ITEMS; ++it) if (val[it]) st_wt_sys(dstb[it] + po, v[it]);
            asm volatile("s_waitcnt vmcnt(0)" ::: "memory");
            __syncthreads();
            if (rcnt) __hip_atomic_fetch_add(rcnt + (size_t)panel * NR * 32, 1u, __ATOMIC_RELAXED, __HIP_MEMORY_SCOPE_SYSTEM);
        }
        if (dead && lane == 0) ctl[1] = 1u;
        asm volatile("s_waitcnt vmcnt(0)" ::: "memory");
        __syncthreads();
        e0 = 4 * NGX + xi; estep = 128; eend = 128;
    } else {
        const int gi = xi - NCX;
        e0 = gi; estep = NGX; eend = 4 * NGX;
    }
    const int extra_e = 4 * NGX + NCX + (xi - NCX);
    auto next_entry = [&](int r) -> int {
        const int e = e0 + r * estep;
        if (e < eend) return e;
        if (xi >= NCX && r == 4 && extra_e < 128) return extra_e;
        return -1;
    };

    const int wr = wid >> 2, wc = wid & 3, fr = lane & 15, fq = lane >> 4;
    const float scale = P.sx[0] * P.sw[0];
    const unsigned voff = (unsigned)tid * 16u;
    const unsigned ldsw = (unsigned)wid * 1024u;
    const int aoff = wr * 4 * 2048 + lane * 16, boff2 = wc * 2 * 2048 + lane * 16;
    auto ready = [&](const Unit& u) {
        if (wid == 0) {
            const int src = lane & 7; const int panel = (lane & 8) ? NPX + u.pn : u.pm;
            const unsigned* c = cnt_ptr(ws, panel, src);
            const unsigned need = (src == rank ? 8u : (unsigned)NC) * call;
            if (!__all(ld_relaxed_sys(c) >= need)) {
                const unsigned long long t0 = __builtin_amdgcn_s_memrealtime();
                while (!__all(ld_relaxed_sys(c) >= need)) {
                    if (__builtin_amdgcn_readfirstlane(__hip_atomic_load(tmo, __ATOMIC_RELAXED, __HIP_MEMORY_SCOPE_AGENT)) == call || __builtin_amdgcn_s_memrealtime() - t0 > DEADLINE_TICKS) {
                        if (lane == 0) { __hip_atomic_store(tmo, call, __ATOMIC_RELAXED, __HIP_MEMORY_SCOPE_AGENT); ctl[1] = 1u; }
                        break; }
                    __builtin_amdgcn_s_sleep(2);
                }
            }
            __builtin_amdgcn_fence(__ATOMIC_ACQUIRE, "");
            asm volatile("s_waitcnt vmcnt(0) lgkmcnt(0)" ::: "memory");
        }
        asm volatile("" ::: "memory"); __builtin_amdgcn_s_barrier(); asm volatile("" ::: "memory");
    };
#define SA(b, h) (((b) * 2 + (h)) * IMG)
#define SB(b, h) ((4 + (b) * 2 + (h)) * IMG)
#define STAGE(bufoff, gbase) do { _Pragma("unroll") for (int _i = 0; _i < 2; ++_i) \
    __builtin_amdgcn_global_load_lds((const unsigned*)((const char*)(gbase) + voff + _i * 8192), (LAS unsigned*)(lds + (bufoff) + ldsw + _i * 8192), 16, 0, 0); } while (0)
#define LDA(dst, b, h) do { _Pragma("unroll") for (int m = 0; m < 4; ++m) _Pragma("unroll") for (int k = 0; k < 2; ++k) dst[m][k] = *(const LAS i32x4*)(lds + SA(b, h) + aoff + m * 2048 + k * 1024); } while (0)
#define LDB(dst, b, h) do { _Pragma("unroll") for (int n = 0; n < 2; ++n) _Pragma("unroll") for (int k = 0; k < 2; ++k) dst[n][k] = *(const LAS i32x4*)(lds + SB(b, h) + boff2 + n * 2048 + k * 1024); } while (0)
#define MMA(ai, bj, At, Bt) do { __builtin_amdgcn_s_setprio(1); _Pragma("unroll") for (int m = 0; m < 4; ++m) _Pragma("unroll") for (int n = 0; n < 2; ++n) _Pragma("unroll") for (int k = 0; k < 2; ++k) \
    acc[ai][bj][m][n] = __builtin_amdgcn_mfma_i32_16x16x64_i8(Bt[n][k], At[m][k], acc[ai][bj][m][n], 0, 0, 0); __builtin_amdgcn_s_setprio(0); } while (0)
#define WAIT_V(n) asm volatile("s_waitcnt vmcnt(" #n ")" ::: "memory")
#define WAIT_L(n) asm volatile("s_waitcnt lgkmcnt(" #n ")" ::: "memory")
#define BAR __builtin_amdgcn_s_barrier()
#define SCHED __builtin_amdgcn_sched_barrier(0)
    int ui = 0;
    const int ef = next_entry(0);
    if (ef < 0) return;
    Unit cur = unit_of(xg, ef), nxt = cur;
    i32x4 acc[2][2][4][2];
#pragma unroll
    for (int a = 0; a < 2; ++a)
#pragma unroll
        for (int b = 0; b < 2; ++b)
#pragma unroll
            for (int m = 0; m < 4; ++m)
#pragma unroll
                for (int n = 0; n < 2; ++n) acc[a][b][m][n] = (i32x4){0, 0, 0, 0};
    i32x4 At[4][2], B0[2][2], B1[2][2];
    const char* cA = (const char*)stg + (size_t)cur.pm * PANEL; const char* cB = (const char*)stg + (size_t)(NPX + cur.pn) * PANEL;
    ready(cur);
    STAGE(SB(0, 0), cB); STAGE(SA(0, 0), cA); STAGE(SB(0, 1), cB + IMG); STAGE(SA(0, 1), cA + IMG);
    if (wr == 1) BAR;
    WAIT_V(4); BAR;
    STAGE(SB(1, 0), cB + KSTEP); STAGE(SA(1, 0), cA + KSTEP); STAGE(SB(1, 1), cB + IMG + KSTEP);
    WAIT_V(6); BAR;
    for (;;) {
        const int en = next_entry(ui + 1);
        const bool has_next = en >= 0;
        if (has_next) nxt = unit_of(xg, en);
        const char* nA = (const char*)stg + (size_t)nxt.pm * PANEL; const char* nB = (const char*)stg + (size_t)(NPX + nxt.pn) * PANEL;
        for (int t = 0; t < NQ; t += 2) {
            const bool last = (t == NQ - 2);
            const char* a1 = cA + (size_t)(t + 1) * KSTEP;
            const char* a2 = last ? nA : cA + (size_t)(t + 2) * KSTEP; const char* b2 = last ? nB : cB + (size_t)(t + 2) * KSTEP;
            const char* a3 = a2 + KSTEP; const char* b3 = b2 + KSTEP;
            if (last && has_next) ready(nxt);
            LDB(B0, 0, 0); SCHED; LDA(At, 0, 0); STAGE(SA(1, 1), a1 + IMG);
            WAIT_L(8); BAR; WAIT_L(0); MMA(0, 0, At, B0); BAR; SCHED;
            LDB(B1, 0, 1); STAGE(SB(0, 0), b2);
            BAR; WAIT_L(0); MMA(0, 1, At, B1); BAR;
            LDA(At, 0, 1); STAGE(SA(0, 0), a2);
            BAR; WAIT_L(0); MMA(1, 0, At, B0); BAR; SCHED;
            STAGE(SB(0, 1), b2 + IMG);
            WAIT_V(6); BAR; MMA(1, 1, At, B1); BAR;
            LDB(B0, 1, 0); SCHED; LDA(At, 1, 0); STAGE(SA(0, 1), a2 + IMG);
            WAIT_L(8); BAR; WAIT_L(0); MMA(0, 0, At, B0); BAR; SCHED;
            LDB(B1, 1, 1); STAGE(SB(1, 0), b3);
            BAR; WAIT_L(0); MMA(0, 1, At, B1); BAR;
            LDA(At, 1, 1); STAGE(SA(1, 0), a3);
            BAR; WAIT_L(0); MMA(1, 0, At, B0); BAR; SCHED;
            STAGE(SB(1, 1), b3 + IMG);
            WAIT_V(6); BAR; MMA(1, 1, At, B1); BAR;
        }
        {
            const bool bad = ctl[1] != 0u;
            const float qn = __builtin_nanf("");
            const int row0 = cur.pm * 256 + wr * 64 + fr, col0 = cur.pn * 256 + wc * 32 + 4 * fq;
#pragma unroll
            for (int ai = 0; ai < 2; ++ai)
#pragma unroll
                for (int m = 0; m < 4; ++m) { float* rowp = P.out + (size_t)(row0 + ai * 128 + m * 16) * N + col0;
#pragma unroll
                    for (int bj = 0; bj < 2; ++bj)
#pragma unroll
                        for (int n = 0; n < 2; ++n) { const i32x4 v = acc[ai][bj][m][n];
                            f32x4 o = {(float)v[0] * scale, (float)v[1] * scale, (float)v[2] * scale, (float)v[3] * scale};
                            if (bad) o = (f32x4){qn, qn, qn, qn};
                            __builtin_nontemporal_store(o, (f32x4*)(rowp + bj * 128 + n * 16)); } }
        }
        if (!has_next) break;
#pragma unroll
        for (int a = 0; a < 2; ++a)
#pragma unroll
            for (int b = 0; b < 2; ++b)
#pragma unroll
                for (int m = 0; m < 4; ++m)
#pragma unroll
                    for (int n = 0; n < 2; ++n) acc[a][b][m][n] = (i32x4){0, 0, 0, 0};
        cur = nxt; cA = nA; cB = nB; ++ui;
    }
    WAIT_V(0);
    if (wr == 0) BAR;
    BAR;
}

extern "C" void kernel_launch(int rank, int nranks, ncclComm_t comm,
                              hipStream_t stream,
                              void* const* d_in, const int* in_sizes, int n_in,
                              void* d_out, int out_size,
                              const int* in_dtypes, int out_dtype,
                              void* const* d_out_all,
                              void* const* d_ws_all, size_t ws_bytes) {
    (void)comm; (void)in_sizes; (void)n_in; (void)out_size; (void)in_dtypes; (void)out_dtype; (void)d_out_all;
    if (nranks != NR || ws_bytes < WS_NEED || (unsigned)rank >= (unsigned)NR) { fprintf(stderr, "unexpected configuration\n"); return; }
    static bool attr_set = false;
    if (!attr_set) {
        HIP_CHECK(hipFuncSetAttribute((const void*)fused_gather_gemm, hipFuncAttributeMaxDynamicSharedMemorySize, LDS_BYTES));
        attr_set = true;
    }
    Params p;
    p.x = (const signed char*)d_in[0]; p.w = (const signed char*)d_in[1];
    p.sx = (const float*)d_in[2]; p.sw = (const float*)d_in[3];
    p.out = (float*)d_out; p.ws_all = d_ws_all; p.rank = rank;
    hipLaunchKernelGGL(fused_gather_gemm, dim3(GRID), dim3(512), LDS_BYTES, stream, p);
    HIP_CHECK(hipGetLastError());
}
```

```cpp
#include <hip/hip_runtime.h>
#include <rccl/rccl.h>
#include <cstdio>
#include <cstdint>

#define HIP_CHECK(expr) do { hipError_t _e = (expr); if (_e != hipSuccess) { \
    fprintf(stderr, "HIP error %s at %s:%d\n", hipGetErrorString(_e), __FILE__, __LINE__); \
    return; } } while (0)

namespace {

#define LAS __attribute__((address_space(3)))
#define GAS __attribute__((address_space(1)))
typedef int i32x4 __attribute__((ext_vector_type(4)));
typedef float f32x4 __attribute__((ext_vector_type(4)));

constexpr int M = 4096, N = 16384, K = 4096, NR = 8;
constexpr int KS = K / NR;
constexpr int IMG = 16384;
constexpr int KSTEP = 2 * IMG;
constexpr int NPX = M / 256, NPW = N / 256, NP = NPX + NPW;
constexpr unsigned KTS = (unsigned)NP * KSTEP;
constexpr unsigned SRCS = 4u * KTS;
constexpr size_t STG_BYTES = (size_t)NR * SRCS;
constexpr int STAGE_BYTES = 8 * IMG;
constexpr int LDS_BYTES = STAGE_BYTES + 1024;

constexpr size_t OFF_TICKET = 0, OFF_TMO = 128, OFF_CNTA = 256, OFF_COMMDONE = 384, OFF_OUTDONE = 512, OFF_CNT = 4096, OFF_PCNT = 32768, OFF_DBG = 512u << 10, OFF_STG = 1u << 20;
constexpr size_t WS_NEED = OFF_STG + 2 * STG_BYTES;
constexpr unsigned long long DEADLINE_TICKS = 20000000ull;

#define QB 54
#define POLL_SLEEP 8
constexpr int GRID = 256;
constexpr int NC = 16;
constexpr int GW = 14;
constexpr int PSH = 56;
constexpr int NSTEP = 16, NQUAD = 73;

struct Params {
    const signed char* x; const signed char* w; const float* sx; const float* sw; float* out;
    void* const* ws_all; void* const* out_all; int rank;
};

__device__ __forceinline__ GAS unsigned* cnt_ptr(unsigned char* ws, int z, int src) {
    return (GAS unsigned*)(ws + OFF_CNT + (size_t)(z * NR + src) * 128);
}
__device__ __forceinline__ unsigned slice_off(int i, bool& special) {
    int panel, kt; special = false;
    if (i < 124) { kt = i / 31; panel = i - 31 * kt; if (panel == 30) { special = true; panel = NPX + PSH; } }
    else { const int j = i - 124, q = j / GW; kt = q & 3; panel = NPX + GW * (1 + (q >> 2)) + (j - GW * q); }
    return (unsigned)kt * KTS + (unsigned)panel * (unsigned)KSTEP;
}
__device__ __forceinline__ int zof(int q) {
    if (q < 31) return q <= 7 ? 0 : q <= 15 ? 1 : q <= 23 ? 2 : 3;
    const int j = q - 31, h = j / 7, e = j - 7 * h;
    return 4 + 2 * h + (e > 3 ? 1 : 0);
}
__device__ __forceinline__ int last_quad(int z) { return z < 3 ? 8 * z + 7 : z == 3 ? 30 : 31 + 7 * ((z - 4) >> 1) + (((z - 4) & 1) ? 6 : 3); }
__device__ __forceinline__ int next_quad(int q, int g) {
    if (g == 0) return (q < 30 || q >= QB) ? q + 2 : q + 1;
    return q == 29 ? QB + 1 : q + 2;
}
__device__ __forceinline__ void st_wt_sys(void* p, i32x4 v) {
    asm volatile("global_store_dwordx4 %0, %1, off sc0 sc1\n\ts_nop 1" :: "v"(p), "v"(v) : "memory");
}
__device__ __forceinline__ unsigned ld_relaxed_sys(const GAS unsigned* p) { return __hip_atomic_load(p, __ATOMIC_RELAXED, __HIP_MEMORY_SCOPE_SYSTEM); }

struct Unit { int pm, pn, zb; };
__device__ __forceinline__ unsigned koff(int u, int rank) {
    if (u < 4) return (unsigned)(rank * 4 + u) * KTS;
    const int v = u - 4, kt = (v * 37) >> 8, s = (rank + 1 + (v - 7 * kt)) & 7;
    return (unsigned)(s * 4 + kt) * KTS;
}
__device__ __forceinline__ int need_z(const Unit& U, int u) {
    if (u < 4) return -1;
    const int kt = ((u - 4) * 37) >> 8;
    return U.zb + kt;
}

}

#define DBG_STAMP(i) do {} while (0)
#define DBG_PANEL(s) do {} while (0)
#define DBG_CYC(i) do {} while (0)
__global__ __launch_bounds__(512, 2) void fused_gather_gemm(Params P) {
    extern __shared__ __attribute__((aligned(16))) unsigned char shm[];
    LAS unsigned char* lds = (LAS unsigned char*)shm;
    LAS unsigned* ctl = (LAS unsigned*)(lds + STAGE_BYTES);
    const int tid = threadIdx.x, wid = __builtin_amdgcn_readfirstlane(tid >> 6), lane = tid & 63;
    const int rank = P.rank;
    unsigned char* ws = (unsigned char*)P.ws_all[rank];
    GAS unsigned* tmo = (GAS unsigned*)(ws + OFF_TMO);
    DBG_STAMP(0);
    if (tid == 0) {
        const unsigned t = __hip_atomic_fetch_add((GAS unsigned*)(ws + OFF_TICKET), 1u, __ATOMIC_RELAXED, __HIP_MEMORY_SCOPE_AGENT);
        ctl[0] = t / gridDim.x + 1u; ctl[1] = 0u;
    }
    __syncthreads();
    const unsigned call = __builtin_amdgcn_readfirstlane(ctl[0]);
    unsigned char* stg = ws + OFF_STG + (size_t)(call & 1u) * STG_BYTES;

    {
        const int img = wid * GRID + (int)blockIdx.x;
        if (img < NP * 8) {
            const int panel = img >> 3, sub = img & 7, kt = sub >> 1, half = sub & 1;
            if (panel < NPX) {
                unsigned char* dst = stg + (size_t)rank * SRCS + (size_t)kt * KTS + (size_t)panel * KSTEP + (size_t)half * IMG;
                const signed char* src = P.x + (size_t)(panel * 256 + half * 128 + (lane & 15)) * KS + kt * 128 + (lane >> 4) * 16;
                i32x4 v[16];
#pragma unroll
                for (int i = 0; i < 16; ++i) v[i] = *(const i32x4*)(src + (size_t)(i >> 1) * 16 * KS + (i & 1) * 64);
#pragma unroll
                for (int i = 0; i < 16; ++i) st_wt_sys(dst + i * 1024 + lane * 16, v[i]);
            } else {
                const int g = lane >> 4, c16 = lane & 15;
                const signed char* src = P.w + (size_t)(kt * 128 + half * 64 + 16 * g) * N + (panel - NPX) * 256 + c16 * 16;
                i32x4 v[16];
#pragma unroll
                for (int j = 0; j < 16; ++j) v[j] = *(const i32x4*)(src + (size_t)j * N);
                LAS unsigned char* tb = lds + wid * IMG;
                i32x4 o[16];
#pragma unroll
                for (int e = 0; e < 4; ++e)
#pragma unroll
                    for (int d = 0; d < 4; ++d) {
                        const unsigned r0 = v[4 * e][d], r1 = v[4 * e + 1][d], r2 = v[4 * e + 2][d], r3 = v[4 * e + 3][d];
                        const unsigned t0 = __builtin_amdgcn_perm(r1, r0, 0x05010400u), t1 = __builtin_amdgcn_perm(r1, r0, 0x07030602u);
                        const unsigned t2 = __builtin_amdgcn_perm(r3, r2, 0x05010400u), t3 = __builtin_amdgcn_perm(r3, r2, 0x07030602u);
                        o[4 * d + 0][e] = (int)__builtin_amdgcn_perm(t2, t0, 0x05040100u);
                        o[4 * d + 1][e] = (int)__builtin_amdgcn_perm(t2, t0, 0x07060302u);
                        o[4 * d + 2][e] = (int)__builtin_amdgcn_perm(t3, t1, 0x05040100u);
                        o[4 * d + 3][e] = (int)__builtin_amdgcn_perm(t3, t1, 0x07060302u);
                    }
#pragma unroll
                for (int c = 0; c < 16; ++c) {
                    const int F = (c & 2) * 4 + (c16 >> 2) * 2 + (c & 1), slot = g * 16 + (c16 & 3) * 4 + (c >> 2);
                    *(LAS i32x4*)(tb + F * 1024 + ((slot ^ ((c16 >> 1) & 3)) * 16)) = o[c];
                }
                asm volatile("s_waitcnt lgkmcnt(0)" ::: "memory");
                __builtin_amdgcn_wave_barrier();
                unsigned char* dw = stg + (size_t)rank * SRCS + (size_t)kt * KTS + (size_t)panel * KSTEP + half * 1024 + lane * 16;
#pragma unroll
                for (int F = 0; F < 16; ++F) {
                    const int swz = ((lane >> 3) & 1) | (((F >> 1) & 1) << 1);
                    const i32x4 t = *(const LAS i32x4*)(tb + F * 1024 + ((lane ^ swz) * 16));
                    st_wt_sys(dw + (F >> 3) * IMG + (F & 7) * 2048, t);
                }
            }
            asm volatile("s_waitcnt vmcnt(0)" ::: "memory");
            if (lane == 0) __hip_atomic_fetch_add((GAS unsigned*)(ws + OFF_PCNT + (size_t)panel * 128), 1u, __ATOMIC_RELAXED, __HIP_MEMORY_SCOPE_AGENT);
        }
    }
    const int xg = (int)blockIdx.x & 7, xi = (int)blockIdx.x >> 3;
    if (wid == 0) {
        int p0, p1;
        if (xi < 4) { p0 = lane; p1 = 64 + (lane & 15); }
        else { p0 = 2 * xg + (xi & 1); p1 = NPX + ((xi - 4) >> 1); }
        const GAS unsigned* c0 = (const GAS unsigned*)(ws + OFF_PCNT + (size_t)p0 * 128); const GAS unsigned* c1 = (const GAS unsigned*)(ws + OFF_PCNT + (size_t)p1 * 128);
        const unsigned need = 8u * call;
        const unsigned long long t0 = __builtin_amdgcn_s_memrealtime();
        while (!__all(__hip_atomic_load(c0, __ATOMIC_RELAXED, __HIP_MEMORY_SCOPE_AGENT) >= need && __hip_atomic_load(c1, __ATOMIC_RELAXED, __HIP_MEMORY_SCOPE_AGENT) >= need)) {
            if (__builtin_amdgcn_readfirstlane(__hip_atomic_load(tmo, __ATOMIC_RELAXED, __HIP_MEMORY_SCOPE_AGENT)) == call || __builtin_amdgcn_s_memrealtime() - t0 > DEADLINE_TICKS) {
                if (lane == 0) { __hip_atomic_store(tmo, call, __ATOMIC_RELAXED, __HIP_MEMORY_SCOPE_AGENT); ctl[1] = 1u; }
                break; }
            __builtin_amdgcn_s_sleep(4);
        }
        __builtin_amdgcn_fence(__ATOMIC_ACQUIRE, "agent");
        asm volatile("s_waitcnt vmcnt(0)" ::: "memory");
        if (blockIdx.x == 0 && lane == 0) __hip_atomic_store((GAS unsigned*)(ws + OFF_CNTA), call, __ATOMIC_RELAXED, __HIP_MEMORY_SCOPE_AGENT);
    }
    __syncthreads();
    DBG_STAMP(1);

    const bool share = xi < 4;
    const int grp = xi >> 1;
    auto comm_phase = [&](const int pass) __attribute__((always_inline)) {
        const int cw = ((int)blockIdx.x & 15) * 8 + wid;
        {
            __amdgpu_buffer_rsrc_t drs[NR - 1];
#pragma unroll
            for (int it = 0; it < NR - 1; ++it) {
                const unsigned peer = (unsigned)(rank + 1 + (it + cw) % (NR - 1)) % (unsigned)NR;
                const unsigned long long ba = (unsigned long long)((unsigned char*)P.ws_all[peer] + OFF_STG + (size_t)(call & 1u) * STG_BYTES + (size_t)rank * SRCS);
                const unsigned lo = __builtin_amdgcn_readfirstlane((unsigned)ba), hi = __builtin_amdgcn_readfirstlane((unsigned)(ba >> 32));
                drs[it] = __builtin_amdgcn_make_buffer_rsrc((void*)(((unsigned long long)hi << 32) | lo), 0, SRCS, 0x00020000);
            }
            const unsigned dvo = (unsigned)((cw & 31) * 1024 + lane * 16);
            GAS unsigned* rcnt = (GAS unsigned*)((unsigned char*)P.ws_all[(unsigned)(rank + 1 + (lane < NR - 1 ? lane : 0)) % (unsigned)NR] + OFF_CNT + (size_t)rank * 128);
            const unsigned char* srcb = stg + (size_t)rank * SRCS + dvo;
            const int sl = cw >> 5;
            int q = grp == 0 ? 0 : (pass == 0 ? 1 : QB + 1);
            const int qend = (grp == 1 && pass == 0) ? 31 : NQUAD;
            int qprev = -1;
            i32x4 va, vb; bool sp;
            va = *(const GAS i32x4*)(srcb + slice_off(4 * q + sl, sp));
#define COMM_STEP(vcur, vnext) do { \
                bool spec, sp2; \
                const int qn = next_quad(q, grp); \
                const unsigned po = slice_off(4 * q + sl, spec); \
                vnext = *(const GAS i32x4*)(srcb + slice_off(4 * (qn < qend ? qn : q) + sl, sp2)); \
                __builtin_amdgcn_sched_barrier(0); \
                if (!spec) { _Pragma("unroll") for (int it = 0; it < NR - 1; ++it) __builtin_amdgcn_raw_buffer_store_b128(vcur, drs[it], dvo, po, 17); } \
                else { _Pragma("unroll") for (int it = 0; it < NR - 1; ++it) { \
                    const unsigned pp = po + (unsigned)((rank + 1 + (it + cw) % (NR - 1)) & 7) * (unsigned)KSTEP; \
                    const i32x4 t = *(const GAS i32x4*)(srcb + pp); \
                    __builtin_amdgcn_raw_buffer_store_b128(t, drs[it], dvo + pp, 0, 17); } } \
                asm volatile("s_waitcnt vmcnt(8)" ::: "memory");        \
                __builtin_amdgcn_s_barrier(); asm volatile("" ::: "memory"); \
                if (qprev >= 0 && wid == 0) { const int z = zof(qprev); \
                    if (z != zof(q)) { if (lane < NR - 1) __hip_atomic_fetch_add(rcnt + (size_t)z * NR * 32, 1u, __ATOMIC_RELAXED, __HIP_MEMORY_SCOPE_SYSTEM); DBG_PANEL(z); } } \
                qprev = q; q = qn; } while (0)
            for (;;) {
                COMM_STEP(va, vb);
                if (q >= qend) break;
                COMM_STEP(vb, va);
                if (q >= qend) break;
            }
            asm volatile("s_waitcnt vmcnt(0)" :: "v"(va), "v"(vb) : "memory");
            __builtin_amdgcn_s_barrier(); asm volatile("" ::: "memory");
            if (wid == 0) { const int z = zof(qprev); if (lane < NR - 1) __hip_atomic_fetch_add(rcnt + (size_t)z * NR * 32, 1u, __ATOMIC_RELAXED, __HIP_MEMORY_SCOPE_SYSTEM); DBG_PANEL(z); }
        }
        if (grp == 0 || pass == 1) {
            DBG_STAMP(2);
            if (wid == 0) {
                const GAS unsigned* sd = (const GAS unsigned*)(ws + OFF_COMMDONE); const unsigned need = 16u * call;
                const unsigned long long t0 = __builtin_amdgcn_s_memrealtime();
                while ((unsigned)__builtin_amdgcn_readfirstlane(__hip_atomic_load(sd, __ATOMIC_RELAXED, __HIP_MEMORY_SCOPE_AGENT)) < need) {
                    if (__builtin_amdgcn_readfirstlane(__hip_atomic_load(tmo, __ATOMIC_RELAXED, __HIP_MEMORY_SCOPE_AGENT)) == call || __builtin_amdgcn_s_memrealtime() - t0 > DEADLINE_TICKS) {
                        if (lane == 0) { __hip_atomic_store(tmo, call, __ATOMIC_RELAXED, __HIP_MEMORY_SCOPE_AGENT); ctl[1] = 1u; }
                        break; }
                    __builtin_amdgcn_s_sleep(8);
                }
                __builtin_amdgcn_fence(__ATOMIC_ACQUIRE, "agent");
                asm volatile("s_waitcnt vmcnt(0) lgkmcnt(0)" ::: "memory");
            }
            __syncthreads();
            const bool bad = ctl[1] != 0u;
            __amdgpu_buffer_rsrc_t ors[NR - 1];
#pragma unroll
            for (int it = 0; it < NR - 1; ++it) {
                const unsigned peer = (unsigned)(rank + 1 + (it + cw) % (NR - 1)) % (unsigned)NR;
                const unsigned long long ba = (unsigned long long)((float*)P.out_all[peer] + (size_t)(PSH + rank) * 256);
                const unsigned lo = __builtin_amdgcn_readfirstlane((unsigned)ba), hi = __builtin_amdgcn_readfirstlane((unsigned)(ba >> 32));
                ors[it] = __builtin_amdgcn_make_buffer_rsrc((void*)(((unsigned long long)hi << 32) | lo), 0, (unsigned)M * N * 4u - (unsigned)(PSH + rank) * 1024u, 0x00020000);
            }
            const unsigned ovo = (unsigned)(grp * 128 + cw) * (unsigned)(N * 4) + (unsigned)lane * 16u;
            const unsigned char* osrc = (const unsigned char*)(P.out + (size_t)(PSH + rank) * 256) + ovo;
            i32x4 va = *(const GAS i32x4*)(osrc), vb;
            const int qnan = 0x7fc00000;
#define OUT_STEP(b, vcur, vnext) do { \
                vnext = *(const GAS i32x4*)(osrc + (size_t)((b) + 1 < NPX ? (b) + 1 : (b)) * 256 * N * 4); \
                __builtin_amdgcn_sched_barrier(0); \
                if (bad) vcur = (i32x4){qnan, qnan, qnan, qnan}; \
                _Pragma("unroll") for (int it = 0; it < NR - 1; ++it) __builtin_amdgcn_raw_buffer_store_b128(vcur, ors[it], ovo + (unsigned)(b) * (unsigned)(256 * N * 4), 0, 17); \
                asm volatile("s_waitcnt vmcnt(8)" ::: "memory"); \
                __builtin_amdgcn_s_barrier(); asm volatile("" ::: "memory"); } while (0)
            for (int b = 0; b < NPX; b += 2) { OUT_STEP(b, va, vb); OUT_STEP(b + 1, vb, va); }
            asm volatile("s_waitcnt vmcnt(0)" :: "v"(va), "v"(vb) : "memory");
            __builtin_amdgcn_s_barrier(); asm volatile("" ::: "memory");
            if (wid == 0) {
                if (lane < NR - 1) __hip_atomic_fetch_add((GAS unsigned*)((unsigned char*)P.ws_all[(unsigned)(rank + 1 + lane) % (unsigned)NR] + OFF_OUTDONE), 1u, __ATOMIC_RELAXED, __HIP_MEMORY_SCOPE_SYSTEM);
                DBG_STAMP(6);
                if (blockIdx.x == 0) {
                    const GAS unsigned* od = (const GAS unsigned*)(ws + OFF_OUTDONE); const unsigned need = (unsigned)((NR - 1) * 32) * call;
                    const unsigned long long t0 = __builtin_amdgcn_s_memrealtime();
                    while ((unsigned)__builtin_amdgcn_readfirstlane(ld_relaxed_sys(od)) < need) {
                        if (__builtin_amdgcn_readfirstlane(__hip_atomic_load(tmo, __ATOMIC_RELAXED, __HIP_MEMORY_SCOPE_AGENT)) == call || __builtin_amdgcn_s_memrealtime() - t0 > DEADLINE_TICKS) {
                            if (lane == 0) { __hip_atomic_store(tmo, call, __ATOMIC_RELAXED, __HIP_MEMORY_SCOPE_AGENT); P.out[0] = __builtin_nanf(""); }
                            break; }
                        __builtin_amdgcn_s_sleep(8);
                    }
                    __builtin_amdgcn_fence(__ATOMIC_ACQUIRE, "");
                }
            }
            asm volatile("s_waitcnt vmcnt(0)" ::: "memory");
            DBG_STAMP(3);
            return;
        }
        asm volatile("s_waitcnt vmcnt(0)" ::: "memory");
        __syncthreads();
    };
    if (xi < 4) { comm_phase(0); if (grp == 0) return; }
    auto unit_at = [&](int idx, Unit& U) -> bool {
        if (share) { if (idx > 0) return false; U.pm = 2 * xg + (xi & 1); U.pn = PSH + rank; U.zb = 0; return true; }
        if (idx >= 4) return false;
        U.pm = 2 * xg + (xi & 1); U.pn = GW * idx + ((xi - 4) >> 1); U.zb = 4 * idx; return true;
    };

    const int wr = wid >> 2, wc = wid & 3, fr = lane & 15, fq = lane >> 4;
    const float scale = P.sx[0] * P.sw[0];
    const unsigned voff = (unsigned)tid * 16u;
    const unsigned ldsw = (unsigned)wid * 1024u;
    const int aoff = wr * 4 * 2048 + lane * 16, boff2 = wc * 2 * 2048 + lane * 16;
    int zdone = -1;
    auto gate = [&](int zn) {
        if (wid == 0) {
            const int src = lane & 7; int z = zn + (lane >> 3); z = z < NSTEP ? z : NSTEP - 1;
            const GAS unsigned* c = src == rank ? (const GAS unsigned*)(ws + OFF_CNTA) : cnt_ptr(ws, z, src);
            const unsigned need = src == rank ? call : ((z < 4 || last_quad(z) >= QB + 1) ? 2u * NC : (unsigned)NC) * call;
            unsigned long long ok = __ballot(ld_relaxed_sys(c) >= need);
            if ((ok & 0xffull) != 0xffull) {
                const unsigned long long t0 = __builtin_amdgcn_s_memrealtime();
                bool gave_up = false;
                for (;;) {
                    __builtin_amdgcn_s_sleep(POLL_SLEEP);
                    bool mine = true;
                    if (lane < 8) mine = ld_relaxed_sys(c) >= need;
                    if (__all(mine)) break;
                    if (__builtin_amdgcn_readfirstlane(__hip_atomic_load(tmo, __ATOMIC_RELAXED, __HIP_MEMORY_SCOPE_AGENT)) == call || __builtin_amdgcn_s_memrealtime() - t0 > DEADLINE_TICKS) {
                        if (lane == 0) { __hip_atomic_store(tmo, call, __ATOMIC_RELAXED, __HIP_MEMORY_SCOPE_AGENT); ctl[1] = 1u; }
                        gave_up = true; break; }
                }
                ok = gave_up ? ~0ull : 0xffull;
            }
            int nz = 1;
            while (nz < 8 && ((ok >> (8 * nz)) & 0xffull) == 0xffull) ++nz;
            if (lane == 0) ctl[2] = (unsigned)(zn + nz - 1);
            __builtin_amdgcn_fence(__ATOMIC_ACQUIRE, "");
            asm volatile("s_waitcnt vmcnt(0) lgkmcnt(0)" ::: "memory");
        }
        asm volatile("" ::: "memory"); __builtin_amdgcn_s_barrier(); asm volatile("" ::: "memory");
        zdone = (int)__builtin_amdgcn_readfirstlane(*(volatile LAS unsigned*)(ctl + 2));
    };
#define SA(b, h) (((b) * 2 + (h)) * IMG)
#define SB(b, h) ((4 + (b) * 2 + (h)) * IMG)
#define STAGE(bufoff, gbase) do { _Pragma("unroll") for (int _i = 0; _i < 2; ++_i) \
    __builtin_amdgcn_global_load_lds((const unsigned*)((const char*)(gbase) + voff + _i * 8192), (LAS unsigned*)(lds + (bufoff) + ldsw + _i * 8192), 16, 0, 0); } while (0)
#define LDA(dst, b, h) do { _Pragma("unroll") for (int m = 0; m < 4; ++m) _Pragma("unroll") for (int k = 0; k < 2; ++k) dst[m][k] = *(const LAS i32x4*)(lds + SA(b, h) + aoff + m * 2048 + k * 1024); } while (0)
#define LDB(dst, b, h) do { _Pragma("unroll") for (int n = 0; n < 2; ++n) _Pragma("unroll") for (int k = 0; k < 2; ++k) dst[n][k] = *(const LAS i32x4*)(lds + SB(b, h) + boff2 + n * 2048 + k * 1024); } while (0)
#define MMA(ai, bj, At, Bt) do { __builtin_amdgcn_s_setprio(1); _Pragma("unroll") for (int m = 0; m < 4; ++m) _Pragma("unroll") for (int n = 0; n < 2; ++n) _Pragma("unroll") for (int k = 0; k < 2; ++k) \
    acc[ai][bj][m][n] = __builtin_amdgcn_mfma_i32_16x16x64_i8(At[m][k], Bt[n][k], acc[ai][bj][m][n], 0, 0, 0); __builtin_amdgcn_s_setprio(0); } while (0)
#define WAIT_V(n) asm volatile("s_waitcnt vmcnt(" #n ")" ::: "memory")
#define WAIT_L(n) asm volatile("s_waitcnt lgkmcnt(" #n ")" ::: "memory")
#define BAR __builtin_amdgcn_s_barrier()
#define SCHED __builtin_amdgcn_sched_barrier(0)
    int ui = 0;
    Unit cur, nxt;
    if (!unit_at(0, cur)) return;
    nxt = cur;
    i32x4 acc[2][2][4][2];
#pragma unroll
    for (int a = 0; a < 2; ++a)
#pragma unroll
        for (int b = 0; b < 2; ++b)
#pragma unroll
            for (int m = 0; m < 4; ++m)
#pragma unroll
                for (int n = 0; n < 2; ++n) acc[a][b][m][n] = (i32x4){0, 0, 0, 0};
    i32x4 At[4][2], B0[2][2], B1[2][2];
    const char* cA = (const char*)stg + (size_t)cur.pm * KSTEP; const char* cB = (const char*)stg + (size_t)(NPX + cur.pn) * KSTEP;
    {
        const unsigned k0 = koff(0, rank), k1 = koff(1, rank);
        STAGE(SB(0, 0), cB + k0); STAGE(SA(0, 0), cA + k0); STAGE(SB(0, 1), cB + k0 + IMG); STAGE(SA(0, 1), cA + k0 + IMG);
        if (wr == 1) BAR;
        WAIT_V(4); BAR;
        STAGE(SB(1, 0), cB + k1); STAGE(SA(1, 0), cA + k1); STAGE(SB(1, 1), cB + k1 + IMG);
        WAIT_V(6); BAR;
    }
    for (;;) {
        const bool has_next = unit_at(ui + 1, nxt);
        const char* nA = (const char*)stg + (size_t)nxt.pm * KSTEP; const char* nB = (const char*)stg + (size_t)(NPX + nxt.pn) * KSTEP;
        DBG_STAMP(4 + 2 * ui);
        for (int u = 0; u < 32; u += 2) {
            const bool last = (u == 30);
            const int zn = last ? -1 : need_z(cur, u + 3);
            if (zn > zdone) gate(zn);
            const char* a1 = cA + koff(u + 1, rank);
            const unsigned k2 = koff(last ? 0 : u + 2, rank), k3 = koff(last ? 1 : u + 3, rank);
            const char* a2 = (last ? nA : cA) + k2; const char* b2 = (last ? nB : cB) + k2;
            const char* a3 = (last ? nA : cA) + k3; const char* b3 = (last ? nB : cB) + k3;
            LDB(B0, 0, 0); SCHED; LDA(At, 0, 0); STAGE(SA(1, 1), a1 + IMG);
            WAIT_L(8); BAR; WAIT_L(0); MMA(0, 0, At, B0); BAR; SCHED;
            LDB(B1, 0, 1); STAGE(SB(0, 0), b2);
            BAR; WAIT_L(0); MMA(0, 1, At, B1); BAR;
            LDA(At, 0, 1); STAGE(SA(0, 0), a2);
            BAR; WAIT_L(0); MMA(1, 0, At, B0); BAR; SCHED;
            STAGE(SB(0, 1), b2 + IMG);
            WAIT_V(6); BAR; MMA(1, 1, At, B1); BAR;
            LDB(B0, 1, 0); SCHED; LDA(At, 1, 0); STAGE(SA(0, 1), a2 + IMG);
            WAIT_L(8); BAR; WAIT_L(0); MMA(0, 0, At, B0); BAR; SCHED;
            LDB(B1, 1, 1); STAGE(SB(1, 0), b3);
            BAR; WAIT_L(0); MMA(0, 1, At, B1); BAR;
            LDA(At, 1, 1); STAGE(SA(1, 0), a3);
            BAR; WAIT_L(0); MMA(1, 0, At, B0); BAR; SCHED;
            STAGE(SB(1, 1), b3 + IMG);
            WAIT_V(6); BAR; MMA(1, 1, At, B1); BAR;
        }
        if (wr == 0) BAR;
        DBG_STAMP(5 + 2 * ui);
        {
            const bool bad = *(volatile LAS unsigned*)(ctl + 1) != 0u;
            const float qn = __builtin_nanf("");
            unsigned ovo = (unsigned)(((wr * 64 + 4 * fq) * N + wc * 64 + 4 * fr) * 4); asm volatile("" : "+v"(ovo));
            const size_t toff = (size_t)cur.pm * 256 * N + cur.pn * 256;
            const __amdgpu_buffer_rsrc_t ors = __builtin_amdgcn_make_buffer_rsrc((void*)(P.out + toff), 0, 256u * N * 4u, 0x00020000);
#define EPILOGUE(AUX) do { _Pragma("unroll") for (int ai = 0; ai < 2; ++ai) _Pragma("unroll") for (int m = 0; m < 4; ++m) _Pragma("unroll") for (int r = 0; r < 4; ++r) { \
                f32x4 o = {(float)acc[ai][0][m][0][r] * scale, (float)acc[ai][0][m][1][r] * scale, (float)acc[ai][1][m][0][r] * scale, (float)acc[ai][1][m][1][r] * scale}; \
                if (bad) o = (f32x4){qn, qn, qn, qn}; \
                __builtin_amdgcn_raw_buffer_store_b128(__builtin_bit_cast(i32x4, o), ors, ovo + (unsigned)(ai * 128 + m * 16 + r) * N * 4u, 0, AUX); } } while (0)
            EPILOGUE(2);
            if (share) {
                WAIT_V(0); BAR;
                if (tid == 0) {
                    __builtin_amdgcn_fence(__ATOMIC_RELEASE, "agent");
                    asm volatile("s_waitcnt vmcnt(0)" ::: "memory");
                    __hip_atomic_fetch_add((GAS unsigned*)(ws + OFF_COMMDONE), 1u, __ATOMIC_RELAXED, __HIP_MEMORY_SCOPE_AGENT);
                }
            }
        }
        if (!has_next) break;
#pragma unroll
        for (int a = 0; a < 2; ++a)
#pragma unroll
            for (int b = 0; b < 2; ++b)
#pragma unroll
                for (int m = 0; m < 4; ++m)
#pragma unroll
                    for (int n = 0; n < 2; ++n) acc[a][b][m][n] = (i32x4){0, 0, 0, 0};
        if (wr == 1) BAR;
        cur = nxt; cA = nA; cB = nB; ++ui;
    }
    WAIT_V(0);
    BAR;
    if (!share) { DBG_STAMP(3); return; }
    comm_phase(1);
}

extern "C" void kernel_launch(int rank, int nranks, ncclComm_t comm,
                              hipStream_t stream,
                              void* const* d_in, const int* in_sizes, int n_in,
                              void* d_out, int out_size,
                              const int* in_dtypes, int out_dtype,
                              void* const* d_out_all,
                              void* const* d_ws_all, size_t ws_bytes) {
    (void)comm; (void)in_sizes; (void)n_in; (void)out_size; (void)in_dtypes; (void)out_dtype;
    if (nranks != NR || ws_bytes < WS_NEED || (unsigned)rank >= (unsigned)NR) { fprintf(stderr, "unexpected configuration\n"); return; }
    static bool attr_set[NR] = {};
    if (!attr_set[rank]) {
        HIP_CHECK(hipFuncSetAttribute((const void*)fused_gather_gemm, hipFuncAttributeMaxDynamicSharedMemorySize, LDS_BYTES));
        attr_set[rank] = true;
    }
    Params p;
    p.x = (const signed char*)d_in[0]; p.w = (const signed char*)d_in[1];
    p.sx = (const float*)d_in[2]; p.sw = (const float*)d_in[3];
    p.out = (float*)d_out; p.ws_all = d_ws_all; p.out_all = d_out_all; p.rank = rank;
    hipLaunchKernelGGL(fused_gather_gemm, dim3(GRID), dim3(512), LDS_BYTES, stream, p);
    HIP_CHECK(hipGetLastError());
}
```

```cpp
#include <hip/hip_runtime.h>
#include <rccl/rccl.h>
#include <cstdio>
#include <cstdint>

#define HIP_CHECK(expr) do { hipError_t _e = (expr); if (_e != hipSuccess) { \
    fprintf(stderr, "HIP error %s at %s:%d\n", hipGetErrorString(_e), __FILE__, __LINE__); \
    return; } } while (0)

namespace {

#define LAS __attribute__((address_space(3)))
#define GAS __attribute__((address_space(1)))
typedef int i32x4 __attribute__((ext_vector_type(4)));
typedef float f32x4 __attribute__((ext_vector_type(4)));

constexpr int M = 4096, N = 16384, K = 4096, NR = 8;
constexpr int KS = K / NR;
constexpr int IMG = 16384;
constexpr int KSTEP = 2 * IMG;
constexpr int NPX = M / 256, NPW = N / 256, NP = NPX + NPW;
constexpr unsigned KTS = (unsigned)NP * KSTEP;
constexpr unsigned SRCS = 4u * KTS;
constexpr size_t STG_BYTES = (size_t)NR * SRCS;
constexpr int STAGE_BYTES = 8 * IMG;
constexpr int LDS_BYTES = STAGE_BYTES + 1024;

constexpr size_t OFF_TICKET = 0, OFF_TMO = 128, OFF_CNTA = 256, OFF_SHDONE = 384, OFF_OUTDONE = 512, OFF_WORK = 640, OFF_CNT = 4096, OFF_PCNT = 32768, OFF_DBG = 512u << 10, OFF_STG = 1u << 20;
constexpr size_t WS_NEED = OFF_STG + 2 * STG_BYTES;
constexpr unsigned long long DEADLINE_TICKS = 20000000ull;

#define WARM_LOADS 0
#define POLL_SLEEP 16
constexpr int GRID = 256;
constexpr int NC = 16;
constexpr int GW = 14;
constexpr int PSH = 56;
constexpr int NSTEP = 16, NQUAD = 73;
#define BLIMIT 31
#define NBLK 1
constexpr int NB = NBLK, IPQ = 16 / NB, LQ = NB == 1 ? 4 : 3;
constexpr int XDIRECT = 8 * IPQ;
constexpr int NIN = NQUAD * IPQ, NTOT = NIN + 32 * IPQ;

struct Params {
    const signed char* x; const signed char* w; const float* sx; const float* sw; float* out;
    void* const* ws_all; void* const* out_all; int rank;
};

__device__ __forceinline__ GAS unsigned* cnt_ptr(unsigned char* ws, int z, int src) {
    return (GAS unsigned*)(ws + OFF_CNT + (size_t)(z * NR + src) * 128);
}
__device__ __forceinline__ unsigned slice_off(int i, bool& special) {
    int panel, kt;
    if (i < 32) { kt = i >> 4; panel = i & 15; }
    else if (i < 62) { kt = i >= 47; panel = NPX + (i - 32 - 15 * kt); }
    else if (i < 124) { kt = i / 31; panel = i - 31 * kt; }
    else { const int j = i - 124, q = j / GW; kt = q & 3; panel = NPX + GW * (1 + (q >> 2)) + (j - GW * q); }
    special = i < 124 && panel == NPX + GW;
    if (special) panel = NPX + PSH;
    return (unsigned)kt * KTS + (unsigned)panel * (unsigned)KSTEP;
}
__device__ __forceinline__ int zof(int q) {
    if (q < 31) return q <= 11 ? 0 : q <= 15 ? 1 : q <= 23 ? 2 : 3;
    const int j = q - 31, h = j / 7, e = j - 7 * h;
    return 4 + 2 * h + (e > 3 ? 1 : 0);
}
__device__ __forceinline__ unsigned quads_in_step(int z) { return z == 0 ? 12u : z == 1 ? 4u : z == 2 ? 8u : z == 3 ? 7u : ((z & 1) ? 3u : 4u); }
__device__ __forceinline__ void st_wt_sys(void* p, i32x4 v) {
    asm volatile("global_store_dwordx4 %0, %1, off sc0 sc1\n\ts_nop 1" :: "v"(p), "v"(v) : "memory");
}
__device__ __forceinline__ unsigned ld_relaxed_sys(const GAS unsigned* p) { return __hip_atomic_load(p, __ATOMIC_RELAXED, __HIP_MEMORY_SCOPE_SYSTEM); }

struct Unit { int pm, pn, zb; };
__device__ __forceinline__ unsigned koff(int u, int rank) {
    const int v = u - 4, kt = (v * 37) >> 8, s = (rank + 1 + (v - 7 * kt)) & 7;
    return (unsigned)(u < 4 ? rank * 4 + u : s * 4 + kt) * KTS;
}
__device__ __forceinline__ int need_z(const Unit& U, int u) { return u < 4 ? -1 : U.zb + (((u - 4) * 37) >> 8); }

}

#define DBG_STAMP(i) do {} while (0)
#define DBG_PANEL(s) do {} while (0)
#define DBG_QUAD(t) do {} while (0)
#define DBG_CYC(i) do {} while (0)
__global__ __launch_bounds__(512, 2) void fused_gather_gemm(Params P) {
    extern __shared__ __attribute__((aligned(16))) unsigned char shm[];
    LAS unsigned char* lds = (LAS unsigned char*)shm;
    LAS unsigned* ctl = (LAS unsigned*)(lds + STAGE_BYTES);
    const int tid = threadIdx.x, wid = __builtin_amdgcn_readfirstlane(tid >> 6), lane = tid & 63;
    const int rank = P.rank;
    unsigned char* ws = (unsigned char*)P.ws_all[rank];
    GAS unsigned* tmo = (GAS unsigned*)(ws + OFF_TMO);
    DBG_STAMP(0);
    if (tid == 0) {
        const unsigned t = __hip_atomic_fetch_add((GAS unsigned*)(ws + OFF_TICKET), 1u, __ATOMIC_RELAXED, __HIP_MEMORY_SCOPE_AGENT);
        ctl[0] = t / gridDim.x + 1u; ctl[1] = 0u;
    }
    __syncthreads();
    const unsigned call = __builtin_amdgcn_readfirstlane(ctl[0]);
    unsigned char* stg = ws + OFF_STG + (size_t)(call & 1u) * STG_BYTES;

    {
        const int img = wid * (GRID - 32) + (int)blockIdx.x - 32;
        if (blockIdx.x >= 32 && img < NP * 8) {
            const int panel = img >> 3, sub = img & 7, kt = sub >> 1, half = sub & 1;
            if (panel < NPX) {
                unsigned char* dst = stg + (size_t)rank * SRCS + (size_t)kt * KTS + (size_t)panel * KSTEP + (size_t)half * IMG;
                const signed char* src = P.x + (size_t)(panel * 256 + half * 128 + (lane & 15)) * KS + kt * 128 + (lane >> 4) * 16;
                i32x4 v[16];
#pragma unroll
                for (int i = 0; i < 16; ++i) v[i] = *(const i32x4*)(src + (size_t)(i >> 1) * 16 * KS + (i & 1) * 64);
#pragma unroll
                for (int i = 0; i < 16; ++i) st_wt_sys(dst + i * 1024 + lane * 16, v[i]);
            } else {
                const int g = lane >> 4, c16 = lane & 15;
                const signed char* src = P.w + (size_t)(kt * 128 + half * 64 + 16 * g) * N + (panel - NPX) * 256 + c16 * 16;
                i32x4 v[16];
#pragma unroll
                for (int j = 0; j < 16; ++j) v[j] = *(const i32x4*)(src + (size_t)j * N);
                LAS unsigned char* tb = lds + wid * IMG;
                i32x4 o[16];
#pragma unroll
                for (int e = 0; e < 4; ++e)
#pragma unroll
                    for (int d = 0; d < 4; ++d) {
                        const unsigned r0 = v[4 * e][d], r1 = v[4 * e + 1][d], r2 = v[4 * e + 2][d], r3 = v[4 * e + 3][d];
                        const unsigned t0 = __builtin_amdgcn_perm(r1, r0, 0x05010400u), t1 = __builtin_amdgcn_perm(r1, r0, 0x07030602u);
                        const unsigned t2 = __builtin_amdgcn_perm(r3, r2, 0x05010400u), t3 = __builtin_amdgcn_perm(r3, r2, 0x07030602u);
                        o[4 * d + 0][e] = (int)__builtin_amdgcn_perm(t2, t0, 0x05040100u);
                        o[4 * d + 1][e] = (int)__builtin_amdgcn_perm(t2, t0, 0x07060302u);
                        o[4 * d + 2][e] = (int)__builtin_amdgcn_perm(t3, t1, 0x05040100u);
                        o[4 * d + 3][e] = (int)__builtin_amdgcn_perm(t3, t1, 0x07060302u);
                    }
#pragma unroll
                for (int c = 0; c < 16; ++c) {
                    const int F = (c & 2) * 4 + (c16 >> 2) * 2 + (c & 1), slot = g * 16 + (c16 & 3) * 4 + (c >> 2);
                    *(LAS i32x4*)(tb + F * 1024 + ((slot ^ ((c16 >> 1) & 3)) * 16)) = o[c];
                }
                asm volatile("s_waitcnt lgkmcnt(0)" ::: "memory");
                __builtin_amdgcn_wave_barrier();
                unsigned char* dw = stg + (size_t)rank * SRCS + (size_t)kt * KTS + (size_t)panel * KSTEP + half * 1024 + lane * 16;
#pragma unroll
                for (int F = 0; F < 16; ++F) {
                    const int swz = ((lane >> 3) & 1) | (((F >> 1) & 1) << 1);
                    const i32x4 t = *(const LAS i32x4*)(tb + F * 1024 + ((lane ^ swz) * 16));
                    st_wt_sys(dw + (F >> 3) * IMG + (F & 7) * 2048, t);
                }
            }
            asm volatile("s_waitcnt vmcnt(0)" ::: "memory");
            if (lane == 0) __hip_atomic_fetch_add((GAS unsigned*)(ws + OFF_PCNT + (size_t)panel * 128), 1u, __ATOMIC_RELAXED, __HIP_MEMORY_SCOPE_AGENT);
        }
    }
    const int xg = (int)blockIdx.x & 7, xi = (int)blockIdx.x >> 3;
    auto wait_local = [&]() {
        if (wid == 0) {
            int p0, p1;
            if (xi < 4) { p0 = lane; p1 = 64 + (lane & 15); }
            else { p0 = 2 * xg + (xi & 1); p1 = NPX + ((xi - 4) >> 1); }
            const GAS unsigned* c0 = (const GAS unsigned*)(ws + OFF_PCNT + (size_t)p0 * 128); const GAS unsigned* c1 = (const GAS unsigned*)(ws + OFF_PCNT + (size_t)p1 * 128);
            const unsigned need = 8u * call;
            const unsigned long long t0 = __builtin_amdgcn_s_memrealtime();
            while (!__all(__hip_atomic_load(c0, __ATOMIC_RELAXED, __HIP_MEMORY_SCOPE_AGENT) >= need && __hip_atomic_load(c1, __ATOMIC_RELAXED, __HIP_MEMORY_SCOPE_AGENT) >= need)) {
                if (__builtin_amdgcn_readfirstlane(__hip_atomic_load(tmo, __ATOMIC_RELAXED, __HIP_MEMORY_SCOPE_AGENT)) == call || __builtin_amdgcn_s_memrealtime() - t0 > DEADLINE_TICKS) {
                    if (lane == 0) { __hip_atomic_store(tmo, call, __ATOMIC_RELAXED, __HIP_MEMORY_SCOPE_AGENT); ctl[1] = 1u; }
                    break; }
                __builtin_amdgcn_s_sleep(4);
            }
            __builtin_amdgcn_fence(__ATOMIC_ACQUIRE, "agent");
            asm volatile("s_waitcnt vmcnt(0) lgkmcnt(0)" ::: "memory");
            if (xi < 4 && lane == 0) __hip_atomic_store((GAS unsigned*)(ws + OFF_CNTA), call, __ATOMIC_RELAXED, __HIP_MEMORY_SCOPE_AGENT);
        }
        asm volatile("" ::: "memory"); __builtin_amdgcn_s_barrier(); asm volatile("" ::: "memory");
        DBG_STAMP(1);
    };
    __syncthreads();
    bool loc_ok = false;
    if (xi >= 4) { wait_local(); loc_ok = true; }

    const bool share = xi < 4;
    const int grp = xi >> 1;
    bool drew_last = false;
    auto comm_phase = [&](const int pass) __attribute__((always_inline)) {
        const int rot = (int)blockIdx.x + wid;
        __amdgpu_buffer_rsrc_t drs[NR - 1], ors[NR - 1];
#pragma unroll
        for (int it = 0; it < NR - 1; ++it) {
            const unsigned peer = (unsigned)(rank + 1 + (it + rot) % (NR - 1)) % (unsigned)NR;
            const unsigned long long ba = (unsigned long long)((unsigned char*)P.ws_all[peer] + OFF_STG + (size_t)(call & 1u) * STG_BYTES + (size_t)rank * SRCS);
            drs[it] = __builtin_amdgcn_make_buffer_rsrc((void*)(((unsigned long long)(unsigned)__builtin_amdgcn_readfirstlane((unsigned)(ba >> 32)) << 32) | (unsigned)__builtin_amdgcn_readfirstlane((unsigned)ba)), 0, SRCS, 0x00020000);
            const unsigned long long bo = (unsigned long long)((float*)P.out_all[peer] + (size_t)(PSH + rank) * 256);
            ors[it] = __builtin_amdgcn_make_buffer_rsrc((void*)(((unsigned long long)(unsigned)__builtin_amdgcn_readfirstlane((unsigned)(bo >> 32)) << 32) | (unsigned)__builtin_amdgcn_readfirstlane((unsigned)bo)), 0, (unsigned)M * N * 4u - (unsigned)(PSH + rank) * 1024u, 0x00020000);
        }
        GAS unsigned* rcnt = (GAS unsigned*)((unsigned char*)P.ws_all[(unsigned)(rank + 1 + (lane < NR - 1 ? lane : 0)) % (unsigned)NR] + OFF_CNT + (size_t)rank * 128);
        GAS unsigned* work = (GAS unsigned*)(ws + OFF_WORK);
        const unsigned wbase = (call - 1u) * (unsigned)(NTOT - 64 + 32) - 64u;
        const unsigned char* srcb = stg + (size_t)rank * SRCS + lane * 16;
        const unsigned char* osrc = (const unsigned char*)(P.out + (size_t)(PSH + rank) * 256) + lane * 16;
        const int limit = (grp == 1 && pass == 0) ? BLIMIT * IPQ : NTOT;
        bool sh_ok = false;
        auto item_off = [&](int t, bool& spec) -> unsigned {
            spec = false;
            if (t < NIN) { const int cw = ((t & (IPQ - 1)) * 8 + wid) * NB; return slice_off(4 * (t >> LQ) + (cw >> 5), spec) + (unsigned)(cw & 31) * 1024u; }
            return (unsigned)(((t - NIN) * 8 + wid) * NB) * (unsigned)(N * 4);
        };
        auto wait_sh = [&]() {
            if (wid == 0) {
                const GAS unsigned* sd = (const GAS unsigned*)(ws + OFF_SHDONE); const unsigned need = 16u * call;
                const unsigned long long t0 = __builtin_amdgcn_s_memrealtime();
                while ((unsigned)__builtin_amdgcn_readfirstlane(__hip_atomic_load(sd, __ATOMIC_RELAXED, __HIP_MEMORY_SCOPE_AGENT)) < need) {
                    if (__builtin_amdgcn_readfirstlane(__hip_atomic_load(tmo, __ATOMIC_RELAXED, __HIP_MEMORY_SCOPE_AGENT)) == call || __builtin_amdgcn_s_memrealtime() - t0 > DEADLINE_TICKS) {
                        if (lane == 0) { __hip_atomic_store(tmo, call, __ATOMIC_RELAXED, __HIP_MEMORY_SCOPE_AGENT); ctl[1] = 1u; }
                        break; }
                    __builtin_amdgcn_s_sleep(8);
                }
                __builtin_amdgcn_fence(__ATOMIC_ACQUIRE, "agent");
                asm volatile("s_waitcnt vmcnt(0) lgkmcnt(0)" ::: "memory");
            }
            asm volatile("" ::: "memory"); __builtin_amdgcn_s_barrier(); asm volatile("" ::: "memory");
            sh_ok = true; DBG_STAMP(2);
        };
        auto item_load = [&](int t, i32x4 (&v)[NB]) {
            if (t < XDIRECT) {
                const int cw = ((t & (IPQ - 1)) * 8 + wid) * NB, i = 4 * (t >> LQ) + (cw >> 5), b = cw & 31;
                const signed char* px = P.x + (size_t)((i & 15) * 256 + (b >> 4) * 128 + ((b >> 1) & 7) * 16 + (lane & 15)) * KS + (i >> 4) * 128 + (b & 1) * 64 + (lane >> 4) * 16;
#pragma unroll
                for (int j = 0; j < NB; ++j) v[j] = *(const GAS i32x4*)(px + 64 * j);
                return;
            }
            bool sp; const unsigned o = item_off(t, sp);
            const unsigned char* pp = (t < NIN ? srcb : osrc) + o;
#pragma unroll
            for (int j = 0; j < NB; ++j) v[j] = *(const GAS i32x4*)(pp + j * (t < NIN ? 1024 : N * 4));
        };
        constexpr int NONE = 0x7fffffff;
        int tc, tn, tp = -1;
        if (pass == 0) { tc = (int)blockIdx.x; tn = 32 + (int)blockIdx.x; }
        else {
            if (tid == 0) {
                unsigned a = (unsigned)NONE, b = (unsigned)NONE;
                if (!drew_last) a = __hip_atomic_fetch_add(work, 1u, __ATOMIC_RELAXED, __HIP_MEMORY_SCOPE_AGENT) - wbase;
                if (a < (unsigned)limit) b = __hip_atomic_fetch_add(work, 1u, __ATOMIC_RELAXED, __HIP_MEMORY_SCOPE_AGENT) - wbase;
                ctl[4] = a; ctl[5] = b;
            }
            __syncthreads();
            tc = (int)__builtin_amdgcn_readfirstlane(ctl[4]); tn = (int)__builtin_amdgcn_readfirstlane(ctl[5]);
            __syncthreads();
        }
        if ((tc >= NTOT && tc != NONE) || (tn >= NTOT && tn != NONE)) drew_last = true;
        i32x4 va[NB], vb[NB];
#pragma unroll
        for (int j = 0; j < NB; ++j) { va[j] = (i32x4){0, 0, 0, 0}; vb[j] = va[j]; }
        if (tc < NTOT) { if (tc >= XDIRECT && !loc_ok) { wait_local(); loc_ok = true; } if (tc >= NIN) wait_sh(); item_load(tc, va); }
        const int qnan = 0x7fc00000;
#define COMM_STEP(vcur, vnext, slot) do { \
            unsigned a = (unsigned)NONE; \
            if (tid == 0 && tn < limit) a = __hip_atomic_fetch_add(work, 1u, __ATOMIC_RELAXED, __HIP_MEMORY_SCOPE_AGENT) - wbase; \
            const bool late = tn < NTOT && ((tn >= NIN && !sh_ok) || (tn >= XDIRECT && !loc_ok)); \
            item_load((tn < NTOT && !late) ? tn : tc, vnext); \
            bool spec; const unsigned po = item_off(tc, spec) + (unsigned)lane * 16u; \
            __builtin_amdgcn_sched_barrier(0); \
            if (tc >= NIN) { if (*(volatile LAS unsigned*)(ctl + 1) != 0u) { _Pragma("unroll") for (int j = 0; j < NB; ++j) vcur[j] = (i32x4){qnan, qnan, qnan, qnan}; } \
                _Pragma("unroll") for (int it = 0; it < NR - 1; ++it) _Pragma("unroll") for (int j = 0; j < NB; ++j) __builtin_amdgcn_raw_buffer_store_b128(vcur[j], ors[it], po + (unsigned)(j * N * 4), 0, 17); } \
            else if (!spec) { _Pragma("unroll") for (int it = 0; it < NR - 1; ++it) _Pragma("unroll") for (int j = 0; j < NB; ++j) __builtin_amdgcn_raw_buffer_store_b128(vcur[j], drs[it], po + 1024u * j, 0, 17); } \
            else { _Pragma("unroll") for (int it = 0; it < NR - 1; ++it) _Pragma("unroll") for (int j = 0; j < NB; ++j) { \
                const unsigned pp = po + (unsigned)((rank + 1 + (it + rot) % (NR - 1)) & 7) * (unsigned)KSTEP + 1024u * j; \
                const i32x4 t0 = *(const GAS i32x4*)(stg + (size_t)rank * SRCS + pp); \
                __builtin_amdgcn_raw_buffer_store_b128(t0, drs[it], pp, 0, 17); } } \
            asm volatile("s_waitcnt vmcnt(%0)" :: "i"(8 * NB) : "memory");        \
            if (tid == 0) { ctl[4 + (slot)] = a; asm volatile("s_waitcnt lgkmcnt(0)" ::: "memory"); } \
            __builtin_amdgcn_s_barrier(); asm volatile("" ::: "memory"); \
            if (tp >= 0 && tp < NIN && wid == 0 && lane < NR - 1) __hip_atomic_fetch_add(rcnt + (size_t)zof(tp >> LQ) * NR * 32, 1u, __ATOMIC_RELAXED, __HIP_MEMORY_SCOPE_SYSTEM); \
            if (tp >= 0 && tp < NIN && wid == 0) DBG_PANEL(zof(tp >> LQ)); \
            if (tp >= 0 && wid == 0) DBG_QUAD(NB * tp); \
            if (late) { if (!loc_ok) { wait_local(); loc_ok = true; } if (tn >= NIN) wait_sh(); item_load(tn, vnext); } \
            tp = tc; tc = tn; tn = (int)__builtin_amdgcn_readfirstlane(*(volatile LAS unsigned*)(ctl + 4 + (slot))); \
            if (tn >= NTOT && tn != NONE) drew_last = true; } while (0)
        while (tc < NTOT) {
            COMM_STEP(va, vb, 0);
            if (tc >= NTOT) break;
            COMM_STEP(vb, va, 1);
        }
        asm volatile("s_waitcnt vmcnt(0)" :: "v"(va[0]), "v"(va[NB - 1]), "v"(vb[0]), "v"(vb[NB - 1]) : "memory");
        __builtin_amdgcn_s_barrier(); asm volatile("" ::: "memory");
        if (tp >= 0 && tp < NIN && wid == 0 && lane < NR - 1) __hip_atomic_fetch_add(rcnt + (size_t)zof(tp >> LQ) * NR * 32, 1u, __ATOMIC_RELAXED, __HIP_MEMORY_SCOPE_SYSTEM);
        if (grp == 0 || pass == 1) {
            if (wid == 0) {
                if (lane < NR - 1) __hip_atomic_fetch_add((GAS unsigned*)((unsigned char*)P.ws_all[(unsigned)(rank + 1 + lane) % (unsigned)NR] + OFF_OUTDONE), 1u, __ATOMIC_RELAXED, __HIP_MEMORY_SCOPE_SYSTEM);
                DBG_STAMP(6);
                if (blockIdx.x == 0) {
                    const GAS unsigned* od = (const GAS unsigned*)(ws + OFF_OUTDONE); const unsigned need = (unsigned)((NR - 1) * 32) * call;
                    const unsigned long long t0 = __builtin_amdgcn_s_memrealtime();
                    while ((unsigned)__builtin_amdgcn_readfirstlane(ld_relaxed_sys(od)) < need) {
                        if (__builtin_amdgcn_readfirstlane(__hip_atomic_load(tmo, __ATOMIC_RELAXED, __HIP_MEMORY_SCOPE_AGENT)) == call || __builtin_amdgcn_s_memrealtime() - t0 > DEADLINE_TICKS) {
                            if (lane == 0) { __hip_atomic_store(tmo, call, __ATOMIC_RELAXED, __HIP_MEMORY_SCOPE_AGENT); P.out[0] = __builtin_nanf(""); }
                            break; }
                        __builtin_amdgcn_s_sleep(8);
                    }
                    __builtin_amdgcn_fence(__ATOMIC_ACQUIRE, "");
                }
            }
            asm volatile("s_waitcnt vmcnt(0)" ::: "memory");
            DBG_STAMP(3);
            return;
        }
        asm volatile("s_waitcnt vmcnt(0)" ::: "memory");
        __syncthreads();
    };
    if (xi < 4) { comm_phase(0); if (grp == 0) return; if (!loc_ok) { wait_local(); loc_ok = true; } }
    auto unit_at = [&](int idx, Unit& U) -> bool {
        if (share) { if (idx > 0) return false; U.pm = 2 * xg + (xi & 1); U.pn = PSH + rank; U.zb = 0; return true; }
        if (idx >= 4) return false;
        U.pm = 2 * xg + (xi & 1); U.pn = GW * idx + ((xi - 4) >> 1); U.zb = 4 * idx; return true;
    };

    const int wr = wid >> 2, wc = wid & 3, fr = lane & 15, fq = lane >> 4;
    const float scale = P.sx[0] * P.sw[0];
    const unsigned voff = (unsigned)tid * 16u;
    const unsigned ldsw = (unsigned)wid * 1024u;
    const int aoff = wr * 4 * 2048 + lane * 16, boff2 = wc * 2 * 2048 + lane * 16;
    int zdone = -1;
    auto gate = [&](int zn) {
        if (wid == 0) {
            const int src = lane & 7; int z = zn + (lane >> 3); z = z < NSTEP ? z : NSTEP - 1;
            const GAS unsigned* c = src == rank ? (const GAS unsigned*)(ws + OFF_CNTA) : cnt_ptr(ws, z, src);
            const unsigned need = src == rank ? call : (unsigned)IPQ * quads_in_step(z) * call;
            unsigned long long ok = __ballot(ld_relaxed_sys(c) >= need);
            if ((ok & 0xffull) != 0xffull) {
                const unsigned long long t0 = __builtin_amdgcn_s_memrealtime();
                bool gave_up = false; unsigned warm = blockIdx.x * 32u; (void)warm;
                for (;;) {
                    __builtin_amdgcn_s_sleep(POLL_SLEEP);
                    bool mine = true;
                    if (lane < 8) mine = ld_relaxed_sys(c) >= need;
                    if (__all(mine)) break;
                    if (__builtin_amdgcn_readfirstlane(__hip_atomic_load(tmo, __ATOMIC_RELAXED, __HIP_MEMORY_SCOPE_AGENT)) == call || __builtin_amdgcn_s_memrealtime() - t0 > DEADLINE_TICKS) {
                        if (lane == 0) { __hip_atomic_store(tmo, call, __ATOMIC_RELAXED, __HIP_MEMORY_SCOPE_AGENT); ctl[1] = 1u; }
                        gave_up = true; break; }
                }
                ok = gave_up ? ~0ull : 0xffull;
            }
            int nz = 1;
            while (nz < 8 && ((ok >> (8 * nz)) & 0xffull) == 0xffull) ++nz;
            if (lane == 0) ctl[2] = (unsigned)(zn + nz - 1);
            __builtin_amdgcn_fence(__ATOMIC_ACQUIRE, "");
            asm volatile("s_waitcnt vmcnt(0) lgkmcnt(0)" ::: "memory");
        }
        asm volatile("" ::: "memory"); __builtin_amdgcn_s_barrier(); asm volatile("" ::: "memory");
        zdone = (int)__builtin_amdgcn_readfirstlane(*(volatile LAS unsigned*)(ctl + 2));
    };
#define SA(b, h) (((b) * 2 + (h)) * IMG)
#define SB(b, h) ((4 + (b) * 2 + (h)) * IMG)
#define STAGE(bufoff, gbase) do { _Pragma("unroll") for (int _i = 0; _i < 2; ++_i) \
    __builtin_amdgcn_global_load_lds((const unsigned*)((const char*)(gbase) + voff + _i * 8192), (LAS unsigned*)(lds + (bufoff) + ldsw + _i * 8192), 16, 0, 0); } while (0)
#define LDA(dst, b, h) do { _Pragma("unroll") for (int m = 0; m < 4; ++m) _Pragma("unroll") for (int k = 0; k < 2; ++k) dst[m][k] = *(const LAS i32x4*)(lds + SA(b, h) + aoff + m * 2048 + k * 1024); } while (0)
#define LDB(dst, b, h) do { _Pragma("unroll") for (int n = 0; n < 2; ++n) _Pragma("unroll") for (int k = 0; k < 2; ++k) dst[n][k] = *(const LAS i32x4*)(lds + SB(b, h) + boff2 + n * 2048 + k * 1024); } while (0)
#define MMA(ai, bj, At, Bt) do { __builtin_amdgcn_s_setprio(1); _Pragma("unroll") for (int m = 0; m < 4; ++m) _Pragma("unroll") for (int n = 0; n < 2; ++n) _Pragma("unroll") for (int k = 0; k < 2; ++k) \
    acc[ai][bj][m][n] = __builtin_amdgcn_mfma_i32_16x16x64_i8(At[m][k], Bt[n][k], acc[ai][bj][m][n], 0, 0, 0); __builtin_amdgcn_s_setprio(0); } while (0)
#define WAIT_V(n) asm volatile("s_waitcnt vmcnt(" #n ")" ::: "memory")
#define WAIT_L(n) asm volatile("s_waitcnt lgkmcnt(" #n ")" ::: "memory")
#define BAR __builtin_amdgcn_s_barrier()
#define SCHED __builtin_amdgcn_sched_barrier(0)
    int ui = 0;
    Unit cur, nxt;
    if (!unit_at(0, cur)) return;
    nxt = cur;
    i32x4 acc[2][2][4][2];
#pragma unroll
    for (int a = 0; a < 2; ++a)
#pragma unroll
        for (int b = 0; b < 2; ++b)
#pragma unroll
            for (int m = 0; m < 4; ++m)
#pragma unroll
                for (int n = 0; n < 2; ++n) acc[a][b][m][n] = (i32x4){0, 0, 0, 0};
    i32x4 At[4][2], B0[2][2], B1[2][2];
    const char* cA = (const char*)stg + (size_t)cur.pm * KSTEP; const char* cB = (const char*)stg + (size_t)(NPX + cur.pn) * KSTEP;
    {
        const unsigned k0 = koff(0, rank), k1 = koff(1, rank);
        STAGE(SB(0, 0), cB + k0); STAGE(SA(0, 0), cA + k0); STAGE(SB(0, 1), cB + k0 + IMG); STAGE(SA(0, 1), cA + k0 + IMG);
        if (wr == 1) BAR;
        WAIT_V(4); BAR;
        STAGE(SB(1, 0), cB + k1); STAGE(SA(1, 0), cA + k1); STAGE(SB(1, 1), cB + k1 + IMG);
        WAIT_V(6); BAR;
    }
    for (;;) {
        const bool has_next = unit_at(ui + 1, nxt);
        const char* nA = (const char*)stg + (size_t)nxt.pm * KSTEP; const char* nB = (const char*)stg + (size_t)(NPX + nxt.pn) * KSTEP;
        DBG_STAMP(4 + 2 * ui);
        for (int u = 0; u < 32; u += 2) {
            const bool last = (u == 30);
            const int zn2 = last ? -1 : need_z(cur, u + 2), zn3 = last ? -1 : need_z(cur, u + 3);
            const char* a1 = cA + koff(u + 1, rank);
            const unsigned k2 = koff(last ? 0 : u + 2, rank), k3 = koff(last ? 1 : u + 3, rank);
            const char* a2 = (last ? nA : cA) + k2; const char* b2 = (last ? nB : cB) + k2;
            const char* a3 = (last ? nA : cA) + k3; const char* b3 = (last ? nB : cB) + k3;
            LDB(B0, 0, 0); SCHED; LDA(At, 0, 0); STAGE(SA(1, 1), a1 + IMG);
            WAIT_L(8); BAR; WAIT_L(0); MMA(0, 0, At, B0); BAR; SCHED;
            if (zn2 > zdone) gate(zn2);
            LDB(B1, 0, 1); STAGE(SB(0, 0), b2);
            BAR; WAIT_L(0); MMA(0, 1, At, B1); BAR;
            LDA(At, 0, 1); STAGE(SA(0, 0), a2);
            BAR; WAIT_L(0); MMA(1, 0, At, B0); BAR; SCHED;
            STAGE(SB(0, 1), b2 + IMG);
            WAIT_V(6); BAR; MMA(1, 1, At, B1); BAR;
            LDB(B0, 1, 0); SCHED; LDA(At, 1, 0); STAGE(SA(0, 1), a2 + IMG);
            WAIT_L(8); BAR; WAIT_L(0); MMA(0, 0, At, B0); BAR; SCHED;
            if (zn3 > zdone) gate(zn3);
            LDB(B1, 1, 1); STAGE(SB(1, 0), b3);
            BAR; WAIT_L(0); MMA(0, 1, At, B1); BAR;
            LDA(At, 1, 1); STAGE(SA(1, 0), a3);
            BAR; WAIT_L(0); MMA(1, 0, At, B0); BAR; SCHED;
            STAGE(SB(1, 1), b3 + IMG);
            WAIT_V(6); BAR; MMA(1, 1, At, B1); BAR;
        }
        if (wr == 0) BAR;
        DBG_STAMP(5 + 2 * ui);
        {
            const bool bad = *(volatile LAS unsigned*)(ctl + 1) != 0u;
            const float qn = __builtin_nanf("");
            unsigned ovo = (unsigned)(((wr * 64 + 4 * fq) * N + wc * 64 + 4 * fr) * 4); asm volatile("" : "+v"(ovo));
            const size_t toff = (size_t)cur.pm * 256 * N + cur.pn * 256;
            const __amdgpu_buffer_rsrc_t ors = __builtin_amdgcn_make_buffer_rsrc((void*)(P.out + toff), 0, 256u * N * 4u, 0x00020000);
#define EPILOGUE(AUX) do { _Pragma("unroll") for (int ai = 0; ai < 2; ++ai) _Pragma("unroll") for (int m = 0; m < 4; ++m) _Pragma("unroll") for (int r = 0; r < 4; ++r) { \
                f32x4 o = {(float)acc[ai][0][m][0][r] * scale, (float)acc[ai][0][m][1][r] * scale, (float)acc[ai][1][m][0][r] * scale, (float)acc[ai][1][m][1][r] * scale}; \
                if (bad) o = (f32x4){qn, qn, qn, qn}; \
                __builtin_amdgcn_raw_buffer_store_b128(__builtin_bit_cast(i32x4, o), ors, ovo + (unsigned)(ai * 128 + m * 16 + r) * N * 4u, 0, AUX); } } while (0)
            EPILOGUE(2);
            if (share) {
                WAIT_V(0); BAR;
                if (tid == 0) {
                    __builtin_amdgcn_fence(__ATOMIC_RELEASE, "agent");
                    asm volatile("s_waitcnt vmcnt(0)" ::: "memory");
                    __hip_atomic_fetch_add((GAS unsigned*)(ws + OFF_SHDONE), 1u, __ATOMIC_RELAXED, __HIP_MEMORY_SCOPE_AGENT);
                }
            }
        }
        if (!has_next) break;
#pragma unroll
        for (int a = 0; a < 2; ++a)
#pragma unroll
            for (int b = 0; b < 2; ++b)
#pragma unroll
                for (int m = 0; m < 4; ++m)
#pragma unroll
                    for (int n = 0; n < 2; ++n) acc[a][b][m][n] = (i32x4){0, 0, 0, 0};
        if (wr == 1) BAR;
        cur = nxt; cA = nA; cB = nB; ++ui;
    }
    WAIT_V(0);
    BAR;
    if (!share) { DBG_STAMP(3); return; }
    comm_phase(1);
}

extern "C" void kernel_launch(int rank, int nranks, ncclComm_t comm,
                              hipStream_t stream,
                              void* const* d_in, const int* in_sizes, int n_in,
                              void* d_out, int out_size,
                              const int* in_dtypes, int out_dtype,
                              void* const* d_out_all,
                              void* const* d_ws_all, size_t ws_bytes) {
    (void)comm; (void)in_sizes; (void)n_in; (void)out_size; (void)in_dtypes; (void)out_dtype;
    if (nranks != NR || ws_bytes < WS_NEED || (unsigned)rank >= (unsigned)NR) { fprintf(stderr, "unexpected configuration\n"); return; }
    static bool attr_set[NR] = {};
    if (!attr_set[rank]) {
        HIP_CHECK(hipFuncSetAttribute((const void*)fused_gather_gemm, hipFuncAttributeMaxDynamicSharedMemorySize, LDS_BYTES));
        attr_set[rank] = true;
    }
    Params p;
    p.x = (const signed char*)d_in[0]; p.w = (const signed char*)d_in[1];
    p.sx = (const float*)d_in[2]; p.sw = (const float*)d_in[3];
    p.out = (float*)d_out; p.ws_all = d_ws_all; p.out_all = d_out_all; p.rank = rank;
    hipLaunchKernelGGL(fused_gather_gemm, dim3(GRID), dim3(512), LDS_BYTES, stream, p);
    HIP_CHECK(hipGetLastError());
}
```

```cpp
#include <hip/hip_runtime.h>
#include <rccl/rccl.h>
#include <cstdio>
#include <cstdint>

#define HIP_CHECK(expr) do { hipError_t _e = (expr); if (_e != hipSuccess) { \
    fprintf(stderr, "HIP error %s at %s:%d\n", hipGetErrorString(_e), __FILE__, __LINE__); \
    return; } } while (0)

namespace {

#define LAS __attribute__((address_space(3)))
#define GAS __attribute__((address_space(1)))
typedef int i32x4 __attribute__((ext_vector_type(4)));
typedef float f32x4 __attribute__((ext_vector_type(4)));

constexpr int M = 4096, N = 16384, K = 4096, NR = 8;
constexpr int KS = K / NR;
constexpr int IMG = 16384;
constexpr int KSTEP = 2 * IMG;
constexpr int NPX = M / 256, NPW = N / 256, NP = NPX + NPW;
constexpr unsigned KTS = (unsigned)NP * KSTEP;
constexpr unsigned SRCS = 4u * KTS;
constexpr size_t STG_BYTES = (size_t)NR * SRCS;
constexpr int STAGE_BYTES = 8 * IMG;
constexpr int LDS_BYTES = STAGE_BYTES + 1024;

constexpr size_t OFF_TICKET = 0, OFF_TMO = 128, OFF_CNTA = 256, OFF_SHDONE = 384, OFF_OUTDONE = 512, OFF_WORK = 640, OFF_CNT = 4096, OFF_PCNT = 32768, OFF_DBG = 512u << 10, OFF_STG = 1u << 20;
constexpr size_t WS_NEED = OFF_STG + 2 * STG_BYTES;
constexpr unsigned long long DEADLINE_TICKS = 20000000ull;

#define FLUSH_Z 3
#define WARM_LOADS 0
#define POLL_SLEEP 16
constexpr int GRID = 256;
constexpr int NC = 16;
constexpr int GW = 14;
constexpr int PSH = 56;
constexpr int NSTEP = 16, NQUAD = 73;
constexpr int T1 = 23 * 8, T2 = (NQUAD - 8) * 8, NTOT = T2 + 32 * 8;

struct Params {
    const signed char* x; const signed char* w; const float* sx; const float* sw; float* out;
    void* const* ws_all; void* const* out_all; int rank;
};

__device__ __forceinline__ GAS unsigned* cnt_ptr(unsigned char* ws, int z, int src) {
    return (GAS unsigned*)(ws + OFF_CNT + (size_t)(z * NR + src) * 128);
}
__device__ __forceinline__ unsigned slice_off(int i, bool& special) {
    int panel, kt;
    if (i < 32) { kt = i >> 4; panel = i & 15; }
    else if (i < 62) { kt = i >= 47; panel = NPX + (i - 32 - 15 * kt); }
    else if (i < 124) { kt = i / 31; panel = i - 31 * kt; }
    else { const int j = i - 124, q = j / GW; kt = q & 3; panel = NPX + GW * (1 + (q >> 2)) + (j - GW * q); }
    special = i < 124 && panel == NPX + GW;
    if (special) panel = NPX + PSH;
    return (unsigned)kt * KTS + (unsigned)panel * (unsigned)KSTEP;
}
__device__ __forceinline__ int zof(int q) {
    if (q < 31) return q <= 11 ? 0 : q <= 15 ? 1 : q <= 23 ? 2 : 3;
    const int j = q - 31, h = j / 7, e = j - 7 * h;
    return 4 + 2 * h + (e > 3 ? 1 : 0);
}
__device__ __forceinline__ unsigned quads_in_step(int z) { return z == 0 ? 12u : z == 1 ? 4u : z == 2 ? 8u : z == 3 ? 7u : ((z & 1) ? 3u : 4u); }
__device__ __forceinline__ void st_wt_sys(void* p, i32x4 v) {
    asm volatile("global_store_dwordx4 %0, %1, off sc0 sc1\n\ts_nop 1" :: "v"(p), "v"(v) : "memory");
}
__device__ __forceinline__ unsigned ld_relaxed_sys(const GAS unsigned* p) { return __hip_atomic_load(p, __ATOMIC_RELAXED, __HIP_MEMORY_SCOPE_SYSTEM); }

struct Unit { int pm, pn, zb; };
__device__ __forceinline__ unsigned koff(int u, int rank) {
    const int v = u - 4, kt = (v * 37) >> 8, s = (rank + 1 + (v - 7 * kt)) & 7;
    return (unsigned)(u < 4 ? rank * 4 + u : s * 4 + kt) * KTS;
}
__device__ __forceinline__ int need_z(const Unit& U, int u) { return u < 4 ? -1 : U.zb + (((u - 4) * 37) >> 8); }

}

#define DBG_STAMP(i) do {} while (0)
#define DBG_PANEL(s) do {} while (0)
#define DBG_QUAD(t) do {} while (0)
#define DBG_CYC(i) do {} while (0)
__global__ __launch_bounds__(512, 2) void fused_gather_gemm(Params P) {
    extern __shared__ __attribute__((aligned(16))) unsigned char shm[];
    LAS unsigned char* lds = (LAS unsigned char*)shm;
    LAS unsigned* ctl = (LAS unsigned*)(lds + STAGE_BYTES);
    const int tid = threadIdx.x, wid = __builtin_amdgcn_readfirstlane(tid >> 6), lane = tid & 63;
    const int rank = P.rank;
    unsigned char* ws = (unsigned char*)P.ws_all[rank];
    GAS unsigned* tmo = (GAS unsigned*)(ws + OFF_TMO);
    DBG_STAMP(0);
    if (tid == 0) {
        const unsigned t = __hip_atomic_fetch_add((GAS unsigned*)(ws + OFF_TICKET), 1u, __ATOMIC_RELAXED, __HIP_MEMORY_SCOPE_AGENT);
        ctl[0] = t / gridDim.x + 1u; ctl[1] = 0u;
    }
    __syncthreads();
    const unsigned call = __builtin_amdgcn_readfirstlane(ctl[0]);
    unsigned char* stg = ws + OFF_STG + (size_t)(call & 1u) * STG_BYTES;

    {
        const int img = wid * (GRID - 32) + (int)blockIdx.x - 32;
        if (blockIdx.x >= 32 && img < NP * 8) {
            const int panel = img >> 3, sub = img & 7, kt = sub >> 1, half = sub & 1;
            if (panel < NPX) {
                unsigned char* dst = stg + (size_t)rank * SRCS + (size_t)kt * KTS + (size_t)panel * KSTEP + (size_t)half * IMG;
                const signed char* src = P.x + (size_t)(panel * 256 + half * 128 + (lane & 15)) * KS + kt * 128 + (lane >> 4) * 16;
                i32x4 v[16];
#pragma unroll
                for (int i = 0; i < 16; ++i) v[i] = *(const i32x4*)(src + (size_t)(i >> 1) * 16 * KS + (i & 1) * 64);
#pragma unroll
                for (int i = 0; i < 16; ++i) st_wt_sys(dst + i * 1024 + lane * 16, v[i]);
            } else {
                const int g = lane >> 4, c16 = lane & 15;
                const signed char* src = P.w + (size_t)(kt * 128 + half * 64 + 16 * g) * N + (panel - NPX) * 256 + c16 * 16;
                i32x4 v[16];
#pragma unroll
                for (int j = 0; j < 16; ++j) v[j] = *(const i32x4*)(src + (size_t)j * N);
                LAS unsigned char* tb = lds + wid * IMG;
                i32x4 o[16];
#pragma unroll
                for (int e = 0; e < 4; ++e)
#pragma unroll
                    for (int d = 0; d < 4; ++d) {
                        const unsigned r0 = v[4 * e][d], r1 = v[4 * e + 1][d], r2 = v[4 * e + 2][d], r3 = v[4 * e + 3][d];
                        const unsigned t0 = __builtin_amdgcn_perm(r1, r0, 0x05010400u), t1 = __builtin_amdgcn_perm(r1, r0, 0x07030602u);
                        const unsigned t2 = __builtin_amdgcn_perm(r3, r2, 0x05010400u), t3 = __builtin_amdgcn_perm(r3, r2, 0x07030602u);
                        o[4 * d + 0][e] = (int)__builtin_amdgcn_perm(t2, t0, 0x05040100u);
                        o[4 * d + 1][e] = (int)__builtin_amdgcn_perm(t2, t0, 0x07060302u);
                        o[4 * d + 2][e] = (int)__builtin_amdgcn_perm(t3, t1, 0x05040100u);
                        o[4 * d + 3][e] = (int)__builtin_amdgcn_perm(t3, t1, 0x07060302u);
                    }
#pragma unroll
                for (int c = 0; c < 16; ++c) {
                    const int F = (c & 2) * 4 + (c16 >> 2) * 2 + (c & 1), slot = g * 16 + (c16 & 3) * 4 + (c >> 2);
                    *(LAS i32x4*)(tb + F * 1024 + ((slot ^ ((c16 >> 1) & 3)) * 16)) = o[c];
                }
                asm volatile("s_waitcnt lgkmcnt(0)" ::: "memory");
                __builtin_amdgcn_wave_barrier();
                unsigned char* dw = stg + (size_t)rank * SRCS + (size_t)kt * KTS + (size_t)panel * KSTEP + half * 1024 + lane * 16;
#pragma unroll
                for (int F = 0; F < 16; ++F) {
                    const int swz = ((lane >> 3) & 1) | (((F >> 1) & 1) << 1);
                    const i32x4 t = *(const LAS i32x4*)(tb + F * 1024 + ((lane ^ swz) * 16));
                    st_wt_sys(dw + (F >> 3) * IMG + (F & 7) * 2048, t);
                }
            }
            asm volatile("s_waitcnt vmcnt(0)" ::: "memory");
            if (lane == 0) __hip_atomic_fetch_add((GAS unsigned*)(ws + OFF_PCNT + (size_t)panel * 128), 1u, __ATOMIC_RELAXED, __HIP_MEMORY_SCOPE_AGENT);
        }
    }
    const int xg = (int)blockIdx.x & 7, xi = (int)blockIdx.x >> 3;
    auto wait_local = [&]() {
        if (wid == 0) {
            int p0, p1;
            if (xi < 4) { p0 = lane; p1 = 64 + (lane & 15); }
            else { p0 = 2 * xg + (xi & 1); p1 = NPX + ((xi - 4) >> 1); }
            const GAS unsigned* c0 = (const GAS unsigned*)(ws + OFF_PCNT + (size_t)p0 * 128); const GAS unsigned* c1 = (const GAS unsigned*)(ws + OFF_PCNT + (size_t)p1 * 128);
            const unsigned need = 8u * call;
            const unsigned long long t0 = __builtin_amdgcn_s_memrealtime();
            while (!__all(__hip_atomic_load(c0, __ATOMIC_RELAXED, __HIP_MEMORY_SCOPE_AGENT) >= need && __hip_atomic_load(c1, __ATOMIC_RELAXED, __HIP_MEMORY_SCOPE_AGENT) >= need)) {
                if (__builtin_amdgcn_readfirstlane(__hip_atomic_load(tmo, __ATOMIC_RELAXED, __HIP_MEMORY_SCOPE_AGENT)) == call || __builtin_amdgcn_s_memrealtime() - t0 > DEADLINE_TICKS) {
                    if (lane == 0) { __hip_atomic_store(tmo, call, __ATOMIC_RELAXED, __HIP_MEMORY_SCOPE_AGENT); ctl[1] = 1u; }
                    break; }
                __builtin_amdgcn_s_sleep(4);
            }
            __builtin_amdgcn_fence(__ATOMIC_ACQUIRE, "agent");
            asm volatile("s_waitcnt vmcnt(0) lgkmcnt(0)" ::: "memory");
            if (xi < 4 && lane == 0) __hip_atomic_store((GAS unsigned*)(ws + OFF_CNTA), call, __ATOMIC_RELAXED, __HIP_MEMORY_SCOPE_AGENT);
        }
        asm volatile("" ::: "memory"); __builtin_amdgcn_s_barrier(); asm volatile("" ::: "memory");
        DBG_STAMP(1);
    };
    __syncthreads();
    bool loc_ok = false;
    if (xi >= 4) { wait_local(); loc_ok = true; }

    const bool share = xi < 4;
    const int grp = xi >> 1;
    bool drew_last = false;
    auto comm_phase = [&](const int pass) __attribute__((always_inline)) {
        const int rot = (int)blockIdx.x + wid;
        __amdgpu_buffer_rsrc_t drs[NR - 1], ors[NR - 1];
#pragma unroll
        for (int it = 0; it < NR - 1; ++it) {
            const unsigned peer = (unsigned)(rank + 1 + (it + rot) % (NR - 1)) % (unsigned)NR;
            const unsigned long long ba = (unsigned long long)((unsigned char*)P.ws_all[peer] + OFF_STG + (size_t)(call & 1u) * STG_BYTES + (size_t)rank * SRCS);
            drs[it] = __builtin_amdgcn_make_buffer_rsrc((void*)(((unsigned long long)(unsigned)__builtin_amdgcn_readfirstlane((unsigned)(ba >> 32)) << 32) | (unsigned)__builtin_amdgcn_readfirstlane((unsigned)ba)), 0, SRCS, 0x00020000);
            const unsigned long long bo = (unsigned long long)((float*)P.out_all[peer] + (size_t)(PSH + rank) * 256);
            ors[it] = __builtin_amdgcn_make_buffer_rsrc((void*)(((unsigned long long)(unsigned)__builtin_amdgcn_readfirstlane((unsigned)(bo >> 32)) << 32) | (unsigned)__builtin_amdgcn_readfirstlane((unsigned)bo)), 0, (unsigned)M * N * 4u - (unsigned)(PSH + rank) * 1024u, 0x00020000);
        }
        GAS unsigned* rcnt = (GAS unsigned*)((unsigned char*)P.ws_all[(unsigned)(rank + 1 + (lane < NR - 1 ? lane : 0)) % (unsigned)NR] + OFF_CNT + (size_t)rank * 128);
        GAS unsigned* work = (GAS unsigned*)(ws + OFF_WORK);
        const unsigned wbase = (call - 1u) * (unsigned)(NTOT - 64 + 32) - 64u;
        const unsigned char* srcb = stg + (size_t)rank * SRCS + lane * 16;
        const unsigned char* osrc = (const unsigned char*)(P.out + (size_t)(PSH + rank) * 256) + lane * 16;
        const int limit = (grp == 1 && pass == 0) ? T1 : NTOT;
        bool sh_ok = false;
        auto item_off = [&](int t, bool& spec) -> unsigned {
            spec = false;
            if (t >= T2) return (unsigned)(((t - T2) * 8 + wid) * 2) * (unsigned)(N * 4);
            const int cw = ((t & 7) * 8 + wid) * 2;
            return slice_off(4 * (8 + (t >> 3)) + (cw >> 5), spec) + (unsigned)(cw & 31) * 1024u;
        };
        auto item_z = [&](int t) { return zof(8 + (t >> 3)); };
        auto wait_sh = [&]() {
            if (wid == 0) {
                const GAS unsigned* sd = (const GAS unsigned*)(ws + OFF_SHDONE); const unsigned need = 16u * call;
                const unsigned long long t0 = __builtin_amdgcn_s_memrealtime();
                while ((unsigned)__builtin_amdgcn_readfirstlane(__hip_atomic_load(sd, __ATOMIC_RELAXED, __HIP_MEMORY_SCOPE_AGENT)) < need) {
                    if (__builtin_amdgcn_readfirstlane(__hip_atomic_load(tmo, __ATOMIC_RELAXED, __HIP_MEMORY_SCOPE_AGENT)) == call || __builtin_amdgcn_s_memrealtime() - t0 > DEADLINE_TICKS) {
                        if (lane == 0) { __hip_atomic_store(tmo, call, __ATOMIC_RELAXED, __HIP_MEMORY_SCOPE_AGENT); ctl[1] = 1u; }
                        break; }
                    __builtin_amdgcn_s_sleep(8);
                }
                __builtin_amdgcn_fence(__ATOMIC_ACQUIRE, "agent");
                asm volatile("s_waitcnt vmcnt(0) lgkmcnt(0)" ::: "memory");
            }
            asm volatile("" ::: "memory"); __builtin_amdgcn_s_barrier(); asm volatile("" ::: "memory");
            sh_ok = true; DBG_STAMP(2);
        };
        auto item_load = [&](int t, i32x4 (&v)[2]) {
            bool sp; const unsigned o = item_off(t, sp);
            const unsigned char* pp = (t < T2 ? srcb : osrc) + o;
            v[0] = *(const GAS i32x4*)pp;
            v[1] = *(const GAS i32x4*)(pp + (t < T2 ? 1024 : N * 4));
        };
        auto signal = [&](int t) {
            if (t == -1 || t >= T2 || wid != 0) return;
            const int z = t == -2 ? 0 : item_z(t);
            if (lane < NR - 1) __hip_atomic_fetch_add(rcnt + (size_t)z * NR * 32, t == -2 ? 2u : 1u, __ATOMIC_RELAXED, __HIP_MEMORY_SCOPE_SYSTEM);
            DBG_PANEL(z);
        };
        constexpr int NONE = 0x7fffffff;
        int tc, tn, tp = -1;
        i32x4 va[2] = {(i32x4){0, 0, 0, 0}, (i32x4){0, 0, 0, 0}}, vb[2] = {va[0], va[1]};
        if (pass == 0) {
            const int i = (int)blockIdx.x, b = wid * 4;
            const signed char* px = P.x + (size_t)((i & 15) * 256 + (b >> 4) * 128 + ((b >> 1) & 7) * 16 + (lane & 15)) * KS + (i >> 4) * 128 + (lane >> 4) * 16;
            i32x4 x4[4];
#pragma unroll
            for (int j = 0; j < 4; ++j) x4[j] = *(const GAS i32x4*)(px + (size_t)(j >> 1) * 16 * KS + (j & 1) * 64);
            const unsigned po = (unsigned)(i >> 4) * KTS + (unsigned)(i & 15) * (unsigned)KSTEP + (unsigned)b * 1024u + (unsigned)lane * 16u;
#define XSTORES(j0) do { _Pragma("unroll") for (int j = j0; j < j0 + 2; ++j) _Pragma("unroll") for (int it = 0; it < NR - 1; ++it) __builtin_amdgcn_raw_buffer_store_b128(x4[j], drs[it], po + 1024u * j, 0, 17); } while (0)
            if (wid != 0) XSTORES(0);
            wait_local(); loc_ok = true;
            tc = (int)blockIdx.x; tn = 32 + (int)blockIdx.x; tp = -2;
            item_load(tc, va);
            __builtin_amdgcn_sched_barrier(0);
            if (wid == 0) XSTORES(0);
            XSTORES(2);
        } else {
            if (tid == 0) {
                unsigned a = (unsigned)NONE, b = (unsigned)NONE;
                if (!drew_last) a = __hip_atomic_fetch_add(work, 1u, __ATOMIC_RELAXED, __HIP_MEMORY_SCOPE_AGENT) - wbase;
                if (a < (unsigned)limit) b = __hip_atomic_fetch_add(work, 1u, __ATOMIC_RELAXED, __HIP_MEMORY_SCOPE_AGENT) - wbase;
                ctl[4] = a; ctl[5] = b;
            }
            __syncthreads();
            tc = (int)__builtin_amdgcn_readfirstlane(ctl[4]); tn = (int)__builtin_amdgcn_readfirstlane(ctl[5]);
            __syncthreads();
        }
        if ((tc >= NTOT && tc != NONE) || (tn >= NTOT && tn != NONE)) drew_last = true;
        if (pass != 0 && tc < NTOT) { if (tc >= T2) wait_sh(); item_load(tc, va); }
        const int qnan = 0x7fc00000;
#define COMM_STEP(vcur, vnext, slot) do { \
            unsigned a = (unsigned)NONE; \
            if (tid == 0 && tn < limit) a = __hip_atomic_fetch_add(work, 1u, __ATOMIC_RELAXED, __HIP_MEMORY_SCOPE_AGENT) - wbase; \
            const bool late = tn < NTOT && tn >= T2 && !sh_ok; \
            item_load((tn < NTOT && !late) ? tn : tc, vnext); \
            bool spec; const unsigned po = item_off(tc, spec) + (unsigned)lane * 16u; \
            __builtin_amdgcn_sched_barrier(0); \
            if (tc >= T2) { if (*(volatile LAS unsigned*)(ctl + 1) != 0u) { vcur[0] = (i32x4){qnan, qnan, qnan, qnan}; vcur[1] = vcur[0]; } \
                _Pragma("unroll") for (int it = 0; it < NR - 1; ++it) { __builtin_amdgcn_raw_buffer_store_b128(vcur[0], ors[it], po, 0, 17); __builtin_amdgcn_raw_buffer_store_b128(vcur[1], ors[it], po + (unsigned)(N * 4), 0, 17); } } \
            else { \
                if (!spec) { _Pragma("unroll") for (int it = 0; it < NR - 1; ++it) { __builtin_amdgcn_raw_buffer_store_b128(vcur[0], drs[it], po, 0, 17); __builtin_amdgcn_raw_buffer_store_b128(vcur[1], drs[it], po + 1024u, 0, 17); } } \
                else { _Pragma("unroll") for (int it = 0; it < NR - 1; ++it) { \
                    const unsigned pp = po + (unsigned)((rank + 1 + (it + rot) % (NR - 1)) & 7) * (unsigned)KSTEP; \
                    const i32x4 t0 = *(const GAS i32x4*)(stg + (size_t)rank * SRCS + pp), t1 = *(const GAS i32x4*)(stg + (size_t)rank * SRCS + pp + 1024); \
                    __builtin_amdgcn_raw_buffer_store_b128(t0, drs[it], pp, 0, 17); __builtin_amdgcn_raw_buffer_store_b128(t1, drs[it], pp + 1024u, 0, 17); } } } \
            const bool flush = tc < T1 && (tn >= T1 || item_z(tn) != item_z(tc)) && item_z(tc) >= FLUSH_Z;     \
            if (flush) asm volatile("s_waitcnt vmcnt(0)" ::: "memory"); \
            else asm volatile("s_waitcnt vmcnt(16)" ::: "memory");        \
            if (tid == 0) { ctl[4 + (slot)] = a; asm volatile("s_waitcnt lgkmcnt(0)" ::: "memory"); } \
            __builtin_amdgcn_s_barrier(); asm volatile("" ::: "memory"); \
            signal(tp); \
            if (flush) { signal(tc); tc = -1; } \
            if (late) { wait_sh(); item_load(tn, vnext); } \
            tp = tc; tc = tn; tn = (int)__builtin_amdgcn_readfirstlane(*(volatile LAS unsigned*)(ctl + 4 + (slot))); \
            if (tn >= NTOT && tn != NONE) drew_last = true; } while (0)
        while (tc < NTOT) {
            COMM_STEP(va, vb, 0);
            if (tc >= NTOT) break;
            COMM_STEP(vb, va, 1);
        }
        asm volatile("s_waitcnt vmcnt(0)" :: "v"(va[0]), "v"(va[1]), "v"(vb[0]), "v"(vb[1]) : "memory");
        __builtin_amdgcn_s_barrier(); asm volatile("" ::: "memory");
        signal(tp);
        if (grp == 0 || pass == 1) {
            if (wid == 0) {
                if (lane < NR - 1) __hip_atomic_fetch_add((GAS unsigned*)((unsigned char*)P.ws_all[(unsigned)(rank + 1 + lane) % (unsigned)NR] + OFF_OUTDONE), 1u, __ATOMIC_RELAXED, __HIP_MEMORY_SCOPE_SYSTEM);
                DBG_STAMP(6);
                if (blockIdx.x == 0) {
                    const GAS unsigned* od = (const GAS unsigned*)(ws + OFF_OUTDONE); const unsigned need = (unsigned)((NR - 1) * 32) * call;
                    const unsigned long long t0 = __builtin_amdgcn_s_memrealtime();
                    while ((unsigned)__builtin_amdgcn_readfirstlane(ld_relaxed_sys(od)) < need) {
                        if (__builtin_amdgcn_readfirstlane(__hip_atomic_load(tmo, __ATOMIC_RELAXED, __HIP_MEMORY_SCOPE_AGENT)) == call || __builtin_amdgcn_s_memrealtime() - t0 > DEADLINE_TICKS) {
                            if (lane == 0) { __hip_atomic_store(tmo, call, __ATOMIC_RELAXED, __HIP_MEMORY_SCOPE_AGENT); P.out[0] = __builtin_nanf(""); }
                            break; }
                        __builtin_amdgcn_s_sleep(8);
                    }
                    __builtin_amdgcn_fence(__ATOMIC_ACQUIRE, "");
                }
            }
            asm volatile("s_waitcnt vmcnt(0)" ::: "memory");
            DBG_STAMP(3);
            return;
        }
        asm volatile("s_waitcnt vmcnt(0)" ::: "memory");
        __syncthreads();
    };
    if (xi < 4) { comm_phase(0); if (grp == 0) return; if (!loc_ok) { wait_local(); loc_ok = true; } }
    auto unit_at = [&](int idx, Unit& U) -> bool {
        if (share) { if (idx > 0) return false; U.pm = 2 * xg + (xi & 1); U.pn = PSH + rank; U.zb = 0; return true; }
        if (idx >= 4) return false;
        U.pm = 2 * xg + (xi & 1); U.pn = GW * idx + ((xi - 4) >> 1); U.zb = 4 * idx; return true;
    };

    const int wr = wid >> 2, wc = wid & 3, fr = lane & 15, fq = lane >> 4;
    const float scale = P.sx[0] * P.sw[0];
    const unsigned voff = (unsigned)tid * 16u;
    const unsigned ldsw = (unsigned)wid * 1024u;
    const int aoff = wr * 4 * 2048 + lane * 16, boff2 = wc * 2 * 2048 + lane * 16;
    int zdone = -1;
    auto gate = [&](int zn) {
        if (wid == 0) {
            const int src = lane & 7; int z = zn + (lane >> 3); z = z < NSTEP ? z : NSTEP - 1;
            const GAS unsigned* c = src == rank ? (const GAS unsigned*)(ws + OFF_CNTA) : cnt_ptr(ws, z, src);
            const unsigned need = src == rank ? call : 8u * quads_in_step(z) * call;
            unsigned long long ok = __ballot(ld_relaxed_sys(c) >= need);
            if ((ok & 0xffull) != 0xffull) {
                const unsigned long long t0 = __builtin_amdgcn_s_memrealtime();
                bool gave_up = false; unsigned warm = blockIdx.x * 32u; (void)warm;
                for (;;) {
                    __builtin_amdgcn_s_sleep(POLL_SLEEP);
                    bool mine = true;
                    if (lane < 8) mine = ld_relaxed_sys(c) >= need;
                    if (__all(mine)) break;
                    if (__builtin_amdgcn_readfirstlane(__hip_atomic_load(tmo, __ATOMIC_RELAXED, __HIP_MEMORY_SCOPE_AGENT)) == call || __builtin_amdgcn_s_memrealtime() - t0 > DEADLINE_TICKS) {
                        if (lane == 0) { __hip_atomic_store(tmo, call, __ATOMIC_RELAXED, __HIP_MEMORY_SCOPE_AGENT); ctl[1] = 1u; }
                        gave_up = true; break; }
                }
                ok = gave_up ? ~0ull : 0xffull;
            }
            int nz = 1;
            while (nz < 8 && ((ok >> (8 * nz)) & 0xffull) == 0xffull) ++nz;
            if (lane == 0) ctl[2] = (unsigned)(zn + nz - 1);
            __builtin_amdgcn_fence(__ATOMIC_ACQUIRE, "");
            asm volatile("s_waitcnt vmcnt(0) lgkmcnt(0)" ::: "memory");
        }
        asm volatile("" ::: "memory"); __builtin_amdgcn_s_barrier(); asm volatile("" ::: "memory");
        zdone = (int)__builtin_amdgcn_readfirstlane(*(volatile LAS unsigned*)(ctl + 2));
    };
#define SA(b, h) (((b) * 2 + (h)) * IMG)
#define SB(b, h) ((4 + (b) * 2 + (h)) * IMG)
#define STAGE(bufoff, gbase) do { _Pragma("unroll") for (int _i = 0; _i < 2; ++_i) \
    __builtin_amdgcn_global_load_lds((const unsigned*)((const char*)(gbase) + voff + _i * 8192), (LAS unsigned*)(lds + (bufoff) + ldsw + _i * 8192), 16, 0, 0); } while (0)
#define LDA(dst, b, h) do { _Pragma("unroll") for (int m = 0; m < 4; ++m) _Pragma("unroll") for (int k = 0; k < 2; ++k) dst[m][k] = *(const LAS i32x4*)(lds + SA(b, h) + aoff + m * 2048 + k * 1024); } while (0)
#define LDB(dst, b, h) do { _Pragma("unroll") for (int n = 0; n < 2; ++n) _Pragma("unroll") for (int k = 0; k < 2; ++k) dst[n][k] = *(const LAS i32x4*)(lds + SB(b, h) + boff2 + n * 2048 + k * 1024); } while (0)
#define MMA(ai, bj, At, Bt) do { __builtin_amdgcn_s_setprio(1); _Pragma("unroll") for (int m = 0; m < 4; ++m) _Pragma("unroll") for (int n = 0; n < 2; ++n) _Pragma("unroll") for (int k = 0; k < 2; ++k) \
    acc[ai][bj][m][n] = __builtin_amdgcn_mfma_i32_16x16x64_i8(At[m][k], Bt[n][k], acc[ai][bj][m][n], 0, 0, 0); __builtin_amdgcn_s_setprio(0); } while (0)
#define WAIT_V(n) asm volatile("s_waitcnt vmcnt(" #n ")" ::: "memory")
#define WAIT_L(n) asm volatile("s_waitcnt lgkmcnt(" #n ")" ::: "memory")
#define BAR __builtin_amdgcn_s_barrier()
#define SCHED __builtin_amdgcn_sched_barrier(0)
    int ui = 0;
    Unit cur, nxt;
    if (!unit_at(0, cur)) return;
    nxt = cur;
    i32x4 acc[2][2][4][2];
#pragma unroll
    for (int a = 0; a < 2; ++a)
#pragma unroll
        for (int b = 0; b < 2; ++b)
#pragma unroll
            for (int m = 0; m < 4; ++m)
#pragma unroll
                for (int n = 0; n < 2; ++n) acc[a][b][m][n] = (i32x4){0, 0, 0, 0};
    i32x4 At[4][2], B0[2][2], B1[2][2];
    const char* cA = (const char*)stg + (size_t)cur.pm * KSTEP; const char* cB = (const char*)stg + (size_t)(NPX + cur.pn) * KSTEP;
    {
        const unsigned k0 = koff(0, rank), k1 = koff(1, rank);
        STAGE(SB(0, 0), cB + k0); STAGE(SA(0, 0), cA + k0); STAGE(SB(0, 1), cB + k0 + IMG); STAGE(SA(0, 1), cA + k0 + IMG);
        if (wr == 1) BAR;
        WAIT_V(4); BAR;
        STAGE(SB(1, 0), cB + k1); STAGE(SA(1, 0), cA + k1); STAGE(SB(1, 1), cB + k1 + IMG);
        WAIT_V(6); BAR;
    }
    for (;;) {
        const bool has_next = unit_at(ui + 1, nxt);
        const char* nA = (const char*)stg + (size_t)nxt.pm * KSTEP; const char* nB = (const char*)stg + (size_t)(NPX + nxt.pn) * KSTEP;
        DBG_STAMP(4 + 2 * ui);
        for (int u = 0; u < 32; u += 2) {
            const bool last = (u == 30);
            const int zn2 = last ? -1 : need_z(cur, u + 2), zn3 = last ? -1 : need_z(cur, u + 3);
            const char* a1 = cA + koff(u + 1, rank);
            const unsigned k2 = koff(last ? 0 : u + 2, rank), k3 = koff(last ? 1 : u + 3, rank);
            const char* a2 = (last ? nA : cA) + k2; const char* b2 = (last ? nB : cB) + k2;
            const char* a3 = (last ? nA : cA) + k3; const char* b3 = (last ? nB : cB) + k3;
            LDB(B0, 0, 0); SCHED; LDA(At, 0, 0); STAGE(SA(1, 1), a1 + IMG);
            WAIT_L(8); BAR; WAIT_L(0); MMA(0, 0, At, B0); BAR; SCHED;
            if (zn2 > zdone) gate(zn2);
            LDB(B1, 0, 1); STAGE(SB(0, 0), b2);
            BAR; WAIT_L(0); MMA(0, 1, At, B1); BAR;
            LDA(At, 0, 1); STAGE(SA(0, 0), a2);
            BAR; WAIT_L(0); MMA(1, 0, At, B0); BAR; SCHED;
            STAGE(SB(0, 1), b2 + IMG);
            WAIT_V(6); BAR; MMA(1, 1, At, B1); BAR;
            LDB(B0, 1, 0); SCHED; LDA(At, 1, 0); STAGE(SA(0, 1), a2 + IMG);
            WAIT_L(8); BAR; WAIT_L(0); MMA(0, 0, At, B0); BAR; SCHED;
            if (zn3 > zdone) gate(zn3);
            LDB(B1, 1, 1); STAGE(SB(1, 0), b3);
            BAR; WAIT_L(0); MMA(0, 1, At, B1); BAR;
            LDA(At, 1, 1); STAGE(SA(1, 0), a3);
            BAR; WAIT_L(0); MMA(1, 0, At, B0); BAR; SCHED;
            STAGE(SB(1, 1), b3 + IMG);
            WAIT_V(6); BAR; MMA(1, 1, At, B1); BAR;
        }
        if (wr == 0) BAR;
        DBG_STAMP(5 + 2 * ui);
        {
            const bool bad = *(volatile LAS unsigned*)(ctl + 1) != 0u;
            const float qn = __builtin_nanf("");
            unsigned ovo = (unsigned)(((wr * 64 + 4 * fq) * N + wc * 64 + 4 * fr) * 4); asm volatile("" : "+v"(ovo));
            const size_t toff = (size_t)cur.pm * 256 * N + cur.pn * 256;
            const __amdgpu_buffer_rsrc_t ors = __builtin_amdgcn_make_buffer_rsrc((void*)(P.out + toff), 0, 256u * N * 4u, 0x00020000);
#define EPILOGUE(AUX) do { _Pragma("unroll") for (int ai = 0; ai < 2; ++ai) _Pragma("unroll") for (int m = 0; m < 4; ++m) _Pragma("unroll") for (int r = 0; r < 4; ++r) { \
                f32x4 o = {(float)acc[ai][0][m][0][r] * scale, (float)acc[ai][0][m][1][r] * scale, (float)acc[ai][1][m][0][r] * scale, (float)acc[ai][1][m][1][r] * scale}; \
                if (bad) o = (f32x4){qn, qn, qn, qn}; \
                __builtin_amdgcn_raw_buffer_store_b128(__builtin_bit_cast(i32x4, o), ors, ovo + (unsigned)(ai * 128 + m * 16 + r) * N * 4u, 0, AUX); } } while (0)
            EPILOGUE(2);
            if (share) {
                WAIT_V(0); BAR;
                if (tid == 0) {
                    __builtin_amdgcn_fence(__ATOMIC_RELEASE, "agent");
                    asm volatile("s_waitcnt vmcnt(0)" ::: "memory");
                    __hip_atomic_fetch_add((GAS unsigned*)(ws + OFF_SHDONE), 1u, __ATOMIC_RELAXED, __HIP_MEMORY_SCOPE_AGENT);
                }
            }
        }
        if (!has_next) break;
#pragma unroll
        for (int a = 0; a < 2; ++a)
#pragma unroll
            for (int b = 0; b < 2; ++b)
#pragma unroll
                for (int m = 0; m < 4; ++m)
#pragma unroll
                    for (int n = 0; n < 2; ++n) acc[a][b][m][n] = (i32x4){0, 0, 0, 0};
        if (wr == 1) BAR;
        cur = nxt; cA = nA; cB = nB; ++ui;
    }
    WAIT_V(0);
    BAR;
    if (!share) { DBG_STAMP(3); return; }
    comm_phase(1);
}

extern "C" void kernel_launch(int rank, int nranks, ncclComm_t comm,
                              hipStream_t stream,
                              void* const* d_in, const int* in_sizes, int n_in,
                              void* d_out, int out_size,
                              const int* in_dtypes, int out_dtype,
                              void* const* d_out_all,
                              void* const* d_ws_all, size_t ws_bytes) {
    (void)comm; (void)in_sizes; (void)n_in; (void)out_size; (void)in_dtypes; (void)out_dtype;
    if (nranks != NR || ws_bytes < WS_NEED || (unsigned)rank >= (unsigned)NR) { fprintf(stderr, "unexpected configuration\n"); return; }
    static bool attr_set[NR] = {};
    if (!attr_set[rank]) {
        HIP_CHECK(hipFuncSetAttribute((const void*)fused_gather_gemm, hipFuncAttributeMaxDynamicSharedMemorySize, LDS_BYTES));
        attr_set[rank] = true;
    }
    Params p;
    p.x = (const signed char*)d_in[0]; p.w = (const signed char*)d_in[1];
    p.sx = (const float*)d_in[2]; p.sw = (const float*)d_in[3];
    p.out = (float*)d_out; p.ws_all = d_ws_all; p.out_all = d_out_all; p.rank = rank;
    hipLaunchKernelGGL(fused_gather_gemm, dim3(GRID), dim3(512), LDS_BYTES, stream, p);
    HIP_CHECK(hipGetLastError());
}
```

```cpp
#include <hip/hip_runtime.h>
#include <rccl/rccl.h>
#include <cstdio>
#include <cstdint>

#define HIP_CHECK(expr) do { hipError_t _e = (expr); if (_e != hipSuccess) { \
    fprintf(stderr, "HIP error %s at %s:%d\n", hipGetErrorString(_e), __FILE__, __LINE__); \
    return; } } while (0)

namespace {

#define LAS __attribute__((address_space(3)))
#define GAS __attribute__((address_space(1)))
typedef int i32x4 __attribute__((ext_vector_type(4)));
typedef float f32x4 __attribute__((ext_vector_type(4)));

constexpr int M = 4096, N = 16384, K = 4096, NR = 8;
constexpr int KS = K / NR;
constexpr int IMG = 16384;
constexpr int KSTEP = 2 * IMG;
constexpr int NPX = M / 256, NPW = N / 256, NP = NPX + NPW;
constexpr unsigned KTS = (unsigned)NP * KSTEP;
constexpr unsigned SRCS = 4u * KTS;
constexpr size_t STG_BYTES = (size_t)NR * SRCS;
constexpr int STAGE_BYTES = 8 * IMG;
constexpr int LDS_BYTES = STAGE_BYTES + 1024;

constexpr size_t OFF_TICKET = 0, OFF_TMO = 128, OFF_CNTA = 256, OFF_SHDONE = 384, OFF_OUTDONE = 512, OFF_WORK = 640, OFF_CNT = 4096, OFF_PCNT = 32768, OFF_DBG = 512u << 10, OFF_STG = 1u << 20;
constexpr size_t WS_NEED = OFF_STG + 2 * STG_BYTES;
constexpr unsigned long long DEADLINE_TICKS = 20000000ull;

#define POLL_SLEEP 16
constexpr int GRID = 256;
constexpr int NC = 16;
constexpr int GW = 14;
constexpr int PSH = 56;
constexpr int NSTEP = 16, NQUAD = 73;
#define BLIMIT 31
#define NBLK 1
constexpr int NB = NBLK, IPQ = 16 / NB, LQ = NB == 1 ? 4 : 3;
constexpr int XDIRECT = 8 * IPQ;
constexpr int NIN = NQUAD * IPQ, NTOT = NIN + 32 * IPQ;

struct Params {
    const signed char* x; const signed char* w; const float* sx; const float* sw; float* out;
    void* const* ws_all; void* const* out_all; int rank;
};

__device__ __forceinline__ GAS unsigned* cnt_ptr(unsigned char* ws, int z, int src) {
    return (GAS unsigned*)(ws + OFF_CNT + (size_t)(z * NR + src) * 128);
}
__device__ __forceinline__ unsigned slice_off(int i, bool& special) {
    int panel, kt;
    if (i < 32) { kt = i >> 4; panel = i & 15; }
    else if (i < 62) { kt = i >= 47; panel = NPX + (i - 32 - 15 * kt); }
    else if (i < 124) { kt = i / 31; panel = i - 31 * kt; }
    else { const int j = i - 124, q = j / GW; kt = q & 3; panel = NPX + GW * (1 + (q >> 2)) + (j - GW * q); }
    special = i < 124 && panel == NPX + GW;
    if (special) panel = NPX + PSH;
    return (unsigned)kt * KTS + (unsigned)panel * (unsigned)KSTEP;
}
__device__ __forceinline__ int zof(int q) {
    if (q < 31) return q <= 11 ? 0 : q <= 15 ? 1 : q <= 23 ? 2 : 3;
    const int j = q - 31, h = j / 7, e = j - 7 * h;
    return 4 + 2 * h + (e > 3 ? 1 : 0);
}
__device__ __forceinline__ unsigned quads_in_step(int z) { return z == 0 ? 12u : z == 1 ? 4u : z == 2 ? 8u : z == 3 ? 7u : ((z & 1) ? 3u : 4u); }
__device__ __forceinline__ void st_wt_sys(void* p, i32x4 v) {
    asm volatile("global_store_dwordx4 %0, %1, off sc0 sc1\n\ts_nop 1" :: "v"(p), "v"(v) : "memory");
}
__device__ __forceinline__ unsigned ld_relaxed_sys(const GAS unsigned* p) { return __hip_atomic_load(p, __ATOMIC_RELAXED, __HIP_MEMORY_SCOPE_SYSTEM); }

struct Unit { int pm, pn, zb; };
__device__ __forceinline__ unsigned koff(int u, int rank) {
    const int v = u - 4, kt = (v * 37) >> 8, s = (rank + 1 + (v - 7 * kt)) & 7;
    return (unsigned)(u < 4 ? rank * 4 + u : s * 4 + kt) * KTS;
}
__device__ __forceinline__ int need_z(const Unit& U, int u) { return u < 4 ? -1 : U.zb + (((u - 4) * 37) >> 8); }

}

#define DBG_STAMP(i) do {} while (0)
#define DBG_PANEL(s) do {} while (0)
#define DBG_QUAD(t) do {} while (0)
#define DBG_CYC(i) do {} while (0)
__global__ __launch_bounds__(512, 2) void fused_gather_gemm(Params P) {
    extern __shared__ __attribute__((aligned(16))) unsigned char shm[];
    LAS unsigned char* lds = (LAS unsigned char*)shm;
    LAS unsigned* ctl = (LAS unsigned*)(lds + STAGE_BYTES);
    const int tid = threadIdx.x, wid = __builtin_amdgcn_readfirstlane(tid >> 6), lane = tid & 63;
    const int rank = P.rank;
    unsigned char* ws = (unsigned char*)P.ws_all[rank];
    GAS unsigned* tmo = (GAS unsigned*)(ws + OFF_TMO);
    DBG_STAMP(0);
    if (tid == 0) {
        const unsigned t = __hip_atomic_fetch_add((GAS unsigned*)(ws + OFF_TICKET), 1u, __ATOMIC_RELAXED, __HIP_MEMORY_SCOPE_AGENT);
        ctl[0] = t / gridDim.x + 1u; ctl[1] = 0u;
    }
    __syncthreads();
    const unsigned call = __builtin_amdgcn_readfirstlane(ctl[0]);
    unsigned char* stg = ws + OFF_STG + (size_t)(call & 1u) * STG_BYTES;

    {
        const int img = wid * (GRID - 32) + (int)blockIdx.x - 32;
        if (blockIdx.x >= 32 && img < NP * 8) {
            const int panel = img >> 3, sub = img & 7, kt = sub >> 1, half = sub & 1;
            if (panel < NPX) {
                unsigned char* dst = stg + (size_t)rank * SRCS + (size_t)kt * KTS + (size_t)panel * KSTEP + (size_t)half * IMG;
                const signed char* src = P.x + (size_t)(panel * 256 + half * 128 + (lane & 15)) * KS + kt * 128 + (lane >> 4) * 16;
                i32x4 v[16];
#pragma unroll
                for (int i = 0; i < 16; ++i) v[i] = *(const i32x4*)(src + (size_t)(i >> 1) * 16 * KS + (i & 1) * 64);
#pragma unroll
                for (int i = 0; i < 16; ++i) st_wt_sys(dst + i * 1024 + lane * 16, v[i]);
            } else {
                const int g = lane >> 4, c16 = lane & 15;
                const signed char* src = P.w + (size_t)(kt * 128 + half * 64 + 16 * g) * N + (panel - NPX) * 256 + c16 * 16;
                i32x4 v[16];
#pragma unroll
                for (int j = 0; j < 16; ++j) v[j] = *(const i32x4*)(src + (size_t)j * N);
                LAS unsigned char* tb = lds + wid * IMG;
                i32x4 o[16];
#pragma unroll
                for (int e = 0; e < 4; ++e)
#pragma unroll
                    for (int d = 0; d < 4; ++d) {
                        const unsigned r0 = v[4 * e][d], r1 = v[4 * e + 1][d], r2 = v[4 * e + 2][d], r3 = v[4 * e + 3][d];
                        const unsigned t0 = __builtin_amdgcn_perm(r1, r0, 0x05010400u), t1 = __builtin_amdgcn_perm(r1, r0, 0x07030602u);
                        const unsigned t2 = __builtin_amdgcn_perm(r3, r2, 0x05010400u), t3 = __builtin_amdgcn_perm(r3, r2, 0x07030602u);
                        o[4 * d + 0][e] = (int)__builtin_amdgcn_perm(t2, t0, 0x05040100u);
                        o[4 * d + 1][e] = (int)__builtin_amdgcn_perm(t2, t0, 0x07060302u);
                        o[4 * d + 2][e] = (int)__builtin_amdgcn_perm(t3, t1, 0x05040100u);
                        o[4 * d + 3][e] = (int)__builtin_amdgcn_perm(t3, t1, 0x07060302u);
                    }
#pragma unroll
                for (int c = 0; c < 16; ++c) {
                    const int F = (c & 2) * 4 + (c16 >> 2) * 2 + (c & 1), slot = g * 16 + (c16 & 3) * 4 + (c >> 2);
                    *(LAS i32x4*)(tb + F * 1024 + ((slot ^ ((c16 >> 1) & 3)) * 16)) = o[c];
                }
                asm volatile("s_waitcnt lgkmcnt(0)" ::: "memory");
                __builtin_amdgcn_wave_barrier();
                unsigned char* dw = stg + (size_t)rank * SRCS + (size_t)kt * KTS + (size_t)panel * KSTEP + half * 1024 + lane * 16;
#pragma unroll
                for (int F = 0; F < 16; ++F) {
                    const int swz = ((lane >> 3) & 1) | (((F >> 1) & 1) << 1);
                    const i32x4 t = *(const LAS i32x4*)(tb + F * 1024 + ((lane ^ swz) * 16));
                    st_wt_sys(dw + (F >> 3) * IMG + (F & 7) * 2048, t);
                }
            }
            asm volatile("s_waitcnt vmcnt(0)" ::: "memory");
            if (lane == 0) __hip_atomic_fetch_add((GAS unsigned*)(ws + OFF_PCNT + (size_t)panel * 128), 1u, __ATOMIC_RELAXED, __HIP_MEMORY_SCOPE_AGENT);
        }
    }
    const int xg = (int)blockIdx.x & 7, xi = (int)blockIdx.x >> 3;
    auto wait_local = [&]() {
        if (wid == 0) {
            int p0, p1;
            if (xi < 4) { p0 = lane; p1 = 64 + (lane & 15); }
            else { p0 = 4 * (xg >> 1) + ((xi - 4) & 3); p1 = NPX + 7 * (xg & 1) + ((xi - 4) >> 2); }
            const GAS unsigned* c0 = (const GAS unsigned*)(ws + OFF_PCNT + (size_t)p0 * 128); const GAS unsigned* c1 = (const GAS unsigned*)(ws + OFF_PCNT + (size_t)p1 * 128);
            const unsigned need = 8u * call;
            const unsigned long long t0 = __builtin_amdgcn_s_memrealtime();
            while (!__all(__hip_atomic_load(c0, __ATOMIC_RELAXED, __HIP_MEMORY_SCOPE_AGENT) >= need && __hip_atomic_load(c1, __ATOMIC_RELAXED, __HIP_MEMORY_SCOPE_AGENT) >= need)) {
                if (__builtin_amdgcn_readfirstlane(__hip_atomic_load(tmo, __ATOMIC_RELAXED, __HIP_MEMORY_SCOPE_AGENT)) == call || __builtin_amdgcn_s_memrealtime() - t0 > DEADLINE_TICKS) {
                    if (lane == 0) { __hip_atomic_store(tmo, call, __ATOMIC_RELAXED, __HIP_MEMORY_SCOPE_AGENT); ctl[1] = 1u; }
                    break; }
                __builtin_amdgcn_s_sleep(4);
            }
            __builtin_amdgcn_fence(__ATOMIC_ACQUIRE, "agent");
            asm volatile("s_waitcnt vmcnt(0) lgkmcnt(0)" ::: "memory");
            if (xi < 4 && lane == 0) __hip_atomic_store((GAS unsigned*)(ws + OFF_CNTA), call, __ATOMIC_RELAXED, __HIP_MEMORY_SCOPE_AGENT);
        }
        asm volatile("" ::: "memory"); __builtin_amdgcn_s_barrier(); asm volatile("" ::: "memory");
        DBG_STAMP(1);
    };
    __syncthreads();
    bool loc_ok = false;
    if (xi >= 4) { wait_local(); loc_ok = true; }

    const bool share = xi < 4;
    const int grp = xi >> 1;
    bool drew_last = false;
    auto comm_phase = [&](const int pass) __attribute__((always_inline)) {
        const int rot = (int)blockIdx.x + wid;
        __amdgpu_buffer_rsrc_t drs[NR - 1], ors[NR - 1];
#pragma unroll
        for (int it = 0; it < NR - 1; ++it) {
            const unsigned peer = (unsigned)(rank + 1 + (it + rot) % (NR - 1)) % (unsigned)NR;
            const unsigned long long ba = (unsigned long long)((unsigned char*)P.ws_all[peer] + OFF_STG + (size_t)(call & 1u) * STG_BYTES + (size_t)rank * SRCS);
            drs[it] = __builtin_amdgcn_make_buffer_rsrc((void*)(((unsigned long long)(unsigned)__builtin_amdgcn_readfirstlane((unsigned)(ba >> 32)) << 32) | (unsigned)__builtin_amdgcn_readfirstlane((unsigned)ba)), 0, SRCS, 0x00020000);
            const unsigned long long bo = (unsigned long long)((float*)P.out_all[peer] + (size_t)(PSH + rank) * 256);
            ors[it] = __builtin_amdgcn_make_buffer_rsrc((void*)(((unsigned long long)(unsigned)__builtin_amdgcn_readfirstlane((unsigned)(bo >> 32)) << 32) | (unsigned)__builtin_amdgcn_readfirstlane((unsigned)bo)), 0, (unsigned)M * N * 4u - (unsigned)(PSH + rank) * 1024u, 0x00020000);
        }
        GAS unsigned* rcnt = (GAS unsigned*)((unsigned char*)P.ws_all[(unsigned)(rank + 1 + (lane < NR - 1 ? lane : 0)) % (unsigned)NR] + OFF_CNT + (size_t)rank * 128);
        GAS unsigned* work = (GAS unsigned*)(ws + OFF_WORK);
        const unsigned wbase = (call - 1u) * (unsigned)(NTOT - 64 + 32) - 64u;
        const unsigned char* srcb = stg + (size_t)rank * SRCS + lane * 16;
        const unsigned char* osrc = (const unsigned char*)(P.out + (size_t)(PSH + rank) * 256) + lane * 16;
        const int limit = (grp == 1 && pass == 0) ? BLIMIT * IPQ : NTOT;
        bool sh_ok = false;
        auto item_off = [&](int t, bool& spec) -> unsigned {
            spec = false;
            if (t < NIN) { const int cw = ((t & (IPQ - 1)) * 8 + wid) * NB; return slice_off(4 * (t >> LQ) + (cw >> 5), spec) + (unsigned)(cw & 31) * 1024u; }
            return (unsigned)(((t - NIN) * 8 + wid) * NB) * (unsigned)(N * 4);
        };
        auto wait_sh = [&]() {
            if (wid == 0) {
                const GAS unsigned* sd = (const GAS unsigned*)(ws + OFF_SHDONE); const unsigned need = 16u * call;
                const unsigned long long t0 = __builtin_amdgcn_s_memrealtime();
                while ((unsigned)__builtin_amdgcn_readfirstlane(__hip_atomic_load(sd, __ATOMIC_RELAXED, __HIP_MEMORY_SCOPE_AGENT)) < need) {
                    if (__builtin_amdgcn_readfirstlane(__hip_atomic_load(tmo, __ATOMIC_RELAXED, __HIP_MEMORY_SCOPE_AGENT)) == call || __builtin_amdgcn_s_memrealtime() - t0 > DEADLINE_TICKS) {
                        if (lane == 0) { __hip_atomic_store(tmo, call, __ATOMIC_RELAXED, __HIP_MEMORY_SCOPE_AGENT); ctl[1] = 1u; }
                        break; }
                    __builtin_amdgcn_s_sleep(8);
                }
                __builtin_amdgcn_fence(__ATOMIC_ACQUIRE, "agent");
                asm volatile("s_waitcnt vmcnt(0) lgkmcnt(0)" ::: "memory");
            }
            asm volatile("" ::: "memory"); __builtin_amdgcn_s_barrier(); asm volatile("" ::: "memory");
            sh_ok = true; DBG_STAMP(2);
        };
        auto item_load = [&](int t, i32x4 (&v)[NB]) {
            if (t < XDIRECT) {
                const int cw = ((t & (IPQ - 1)) * 8 + wid) * NB, i = 4 * (t >> LQ) + (cw >> 5), b = cw & 31;
                const signed char* px = P.x + (size_t)((i & 15) * 256 + (b >> 4) * 128 + ((b >> 1) & 7) * 16 + (lane & 15)) * KS + (i >> 4) * 128 + (b & 1) * 64 + (lane >> 4) * 16;
#pragma unroll
                for (int j = 0; j < NB; ++j) v[j] = *(const GAS i32x4*)(px + 64 * j);
                return;
            }
            bool sp; const unsigned o = item_off(t, sp);
            const unsigned char* pp = (t < NIN ? srcb : osrc) + o;
#pragma unroll
            for (int j = 0; j < NB; ++j) v[j] = *(const GAS i32x4*)(pp + j * (t < NIN ? 1024 : N * 4));
        };
        constexpr int NONE = 0x7fffffff;
        int tc, tn, tp = -1;
        if (pass == 0) { tc = (int)blockIdx.x; tn = 32 + (int)blockIdx.x; }
        else {
            if (tid == 0) {
                unsigned a = (unsigned)NONE, b = (unsigned)NONE;
                if (!drew_last) a = __hip_atomic_fetch_add(work, 1u, __ATOMIC_RELAXED, __HIP_MEMORY_SCOPE_AGENT) - wbase;
                if (a < (unsigned)limit) b = __hip_atomic_fetch_add(work, 1u, __ATOMIC_RELAXED, __HIP_MEMORY_SCOPE_AGENT) - wbase;
                ctl[4] = a; ctl[5] = b;
            }
            __syncthreads();
            tc = (int)__builtin_amdgcn_readfirstlane(ctl[4]); tn = (int)__builtin_amdgcn_readfirstlane(ctl[5]);
            __syncthreads();
        }
        if ((tc >= NTOT && tc != NONE) || (tn >= NTOT && tn != NONE)) drew_last = true;
        i32x4 va[NB], vb[NB];
#pragma unroll
        for (int j = 0; j < NB; ++j) { va[j] = (i32x4){0, 0, 0, 0}; vb[j] = va[j]; }
        if (tc < NTOT) { if (tc >= XDIRECT && !loc_ok) { wait_local(); loc_ok = true; } if (tc >= NIN) wait_sh(); item_load(tc, va); }
        const int qnan = 0x7fc00000;
#define COMM_STEP(vcur, vnext, slot) do { \
            unsigned a = (unsigned)NONE; \
            if (tid == 0 && tn < limit) a = __hip_atomic_fetch_add(work, 1u, __ATOMIC_RELAXED, __HIP_MEMORY_SCOPE_AGENT) - wbase; \
            const bool late = tn < NTOT && ((tn >= NIN && !sh_ok) || (tn >= XDIRECT && !loc_ok)); \
            item_load((tn < NTOT && !late) ? tn : tc, vnext); \
            bool spec; const unsigned po = item_off(tc, spec) + (unsigned)lane * 16u; \
            __builtin_amdgcn_sched_barrier(0); \
            if (tc >= NIN) { if (*(volatile LAS unsigned*)(ctl + 1) != 0u) { _Pragma("unroll") for (int j = 0; j < NB; ++j) vcur[j] = (i32x4){qnan, qnan, qnan, qnan}; } \
                _Pragma("unroll") for (int it = 0; it < NR - 1; ++it) _Pragma("unroll") for (int j = 0; j < NB; ++j) __builtin_amdgcn_raw_buffer_store_b128(vcur[j], ors[it], po + (unsigned)(j * N * 4), 0, 17); } \
            else if (!spec) { _Pragma("unroll") for (int it = 0; it < NR - 1; ++it) _Pragma("unroll") for (int j = 0; j < NB; ++j) __builtin_amdgcn_raw_buffer_store_b128(vcur[j], drs[it], po + 1024u * j, 0, 17); } \
            else { _Pragma("unroll") for (int it = 0; it < NR - 1; ++it) _Pragma("unroll") for (int j = 0; j < NB; ++j) { \
                const unsigned pp = po + (unsigned)((rank + 1 + (it + rot) % (NR - 1)) & 7) * (unsigned)KSTEP + 1024u * j; \
                const i32x4 t0 = *(const GAS i32x4*)(stg + (size_t)rank * SRCS + pp); \
                __builtin_amdgcn_raw_buffer_store_b128(t0, drs[it], pp, 0, 17); } } \
            const bool flush = tc < 31 * IPQ && tn >= 31 * IPQ;      \
            if (flush) asm volatile("s_waitcnt vmcnt(0)" ::: "memory"); \
            else asm volatile("s_waitcnt vmcnt(%0)" :: "i"(8 * NB) : "memory");        \
            if (tid == 0) { ctl[4 + (slot)] = a; asm volatile("s_waitcnt lgkmcnt(0)" ::: "memory"); } \
            __builtin_amdgcn_s_barrier(); asm volatile("" ::: "memory"); \
            if (tp >= 0 && tp < NIN && wid == 0 && lane < NR - 1) __hip_atomic_fetch_add(rcnt + (size_t)zof(tp >> LQ) * NR * 32, 1u, __ATOMIC_RELAXED, __HIP_MEMORY_SCOPE_SYSTEM); \
            if (flush) { if (wid == 0 && lane < NR - 1) __hip_atomic_fetch_add(rcnt + (size_t)zof(tc >> LQ) * NR * 32, 1u, __ATOMIC_RELAXED, __HIP_MEMORY_SCOPE_SYSTEM); tc = -1; } \
            if (tp >= 0 && tp < NIN && wid == 0) DBG_PANEL(zof(tp >> LQ)); \
            if (tp >= 0 && wid == 0) DBG_QUAD(NB * tp); \
            if (late) { if (!loc_ok) { wait_local(); loc_ok = true; } if (tn >= NIN) wait_sh(); item_load(tn, vnext); } \
            tp = tc; tc = tn; tn = (int)__builtin_amdgcn_readfirstlane(*(volatile LAS unsigned*)(ctl + 4 + (slot))); \
            if (tn >= NTOT && tn != NONE) drew_last = true; } while (0)
        while (tc < NTOT) {
            COMM_STEP(va, vb, 0);
            if (tc >= NTOT) break;
            COMM_STEP(vb, va, 1);
        }
        asm volatile("s_waitcnt vmcnt(0)" :: "v"(va[0]), "v"(va[NB - 1]), "v"(vb[0]), "v"(vb[NB - 1]) : "memory");
        __builtin_amdgcn_s_barrier(); asm volatile("" ::: "memory");
        if (tp >= 0 && tp < NIN && wid == 0 && lane < NR - 1) __hip_atomic_fetch_add(rcnt + (size_t)zof(tp >> LQ) * NR * 32, 1u, __ATOMIC_RELAXED, __HIP_MEMORY_SCOPE_SYSTEM);
        if (grp == 0 || pass == 1) {
            if (wid == 0) {
                if (lane < NR - 1) __hip_atomic_fetch_add((GAS unsigned*)((unsigned char*)P.ws_all[(unsigned)(rank + 1 + lane) % (unsigned)NR] + OFF_OUTDONE), 1u, __ATOMIC_RELAXED, __HIP_MEMORY_SCOPE_SYSTEM);
                DBG_STAMP(6);
                if (blockIdx.x == 0) {
                    const GAS unsigned* od = (const GAS unsigned*)(ws + OFF_OUTDONE); const unsigned need = (unsigned)((NR - 1) * 32) * call;
                    const unsigned long long t0 = __builtin_amdgcn_s_memrealtime();
                    while ((unsigned)__builtin_amdgcn_readfirstlane(ld_relaxed_sys(od)) < need) {
                        if (__builtin_amdgcn_readfirstlane(__hip_atomic_load(tmo, __ATOMIC_RELAXED, __HIP_MEMORY_SCOPE_AGENT)) == call || __builtin_amdgcn_s_memrealtime() - t0 > DEADLINE_TICKS) {
                            if (lane == 0) { __hip_atomic_store(tmo, call, __ATOMIC_RELAXED, __HIP_MEMORY_SCOPE_AGENT); P.out[0] = __builtin_nanf(""); }
                            break; }
                        __builtin_amdgcn_s_sleep(8);
                    }
                    __builtin_amdgcn_fence(__ATOMIC_ACQUIRE, "");
                }
            }
            asm volatile("s_waitcnt vmcnt(0)" ::: "memory");
            DBG_STAMP(3);
            return;
        }
        asm volatile("s_waitcnt vmcnt(0)" ::: "memory");
        __syncthreads();
    };
    if (xi < 4) { comm_phase(0); if (grp == 0) return; if (!loc_ok) { wait_local(); loc_ok = true; } }
    auto unit_at = [&](int idx, Unit& U) -> bool {
        if (share) { if (idx > 0) return false; U.pm = 2 * xg + (xi & 1); U.pn = PSH + rank; U.zb = 0; return true; }
        if (idx >= 4) return false;
        U.pm = 4 * (xg >> 1) + ((xi - 4) & 3); U.pn = GW * idx + 7 * (xg & 1) + ((xi - 4) >> 2); U.zb = 4 * idx; return true;
    };

    const int wr = wid >> 2, wc = wid & 3, fr = lane & 15, fq = lane >> 4;
    const float scale = P.sx[0] * P.sw[0];
    const unsigned voff = (unsigned)tid * 16u;
    const unsigned ldsw = (unsigned)wid * 1024u;
    const int aoff = wr * 4 * 2048 + lane * 16, boff2 = wc * 2 * 2048 + lane * 16;
    int zdone = -1;
    auto gate = [&](int zn) {
        if (wid == 0) {
            const int src = lane & 7; int z = zn + (lane >> 3); z = z < NSTEP ? z : NSTEP - 1;
            const GAS unsigned* c = src == rank ? (const GAS unsigned*)(ws + OFF_CNTA) : cnt_ptr(ws, z, src);
            const unsigned need = src == rank ? call : (unsigned)IPQ * quads_in_step(z) * call;
            unsigned long long ok = __ballot(ld_relaxed_sys(c) >= need);
            if ((ok & 0xffull) != 0xffull) {
                const unsigned long long t0 = __builtin_amdgcn_s_memrealtime();
                bool gave_up = false;
                for (;;) {
                    __builtin_amdgcn_s_sleep(POLL_SLEEP);
                    bool mine = true;
                    if (lane < 8) mine = ld_relaxed_sys(c) >= need;
                    if (__all(mine)) break;
                    if (__builtin_amdgcn_readfirstlane(__hip_atomic_load(tmo, __ATOMIC_RELAXED, __HIP_MEMORY_SCOPE_AGENT)) == call || __builtin_amdgcn_s_memrealtime() - t0 > DEADLINE_TICKS) {
                        if (lane == 0) { __hip_atomic_store(tmo, call, __ATOMIC_RELAXED, __HIP_MEMORY_SCOPE_AGENT); ctl[1] = 1u; }
                        gave_up = true; break; }
                }
                ok = gave_up ? ~0ull : 0xffull;
            }
            int nz = 1;
            while (nz < 8 && ((ok >> (8 * nz)) & 0xffull) == 0xffull) ++nz;
            if (lane == 0) ctl[2] = (unsigned)(zn + nz - 1);
            __builtin_amdgcn_fence(__ATOMIC_ACQUIRE, "");
            asm volatile("s_waitcnt vmcnt(0) lgkmcnt(0)" ::: "memory");
        }
        asm volatile("" ::: "memory"); __builtin_amdgcn_s_barrier(); asm volatile("" ::: "memory");
        zdone = (int)__builtin_amdgcn_readfirstlane(*(volatile LAS unsigned*)(ctl + 2));
    };
#define SA(b, h) (((b) * 2 + (h)) * IMG)
#define SB(b, h) ((4 + (b) * 2 + (h)) * IMG)
#define STAGE(bufoff, gbase) do { _Pragma("unroll") for (int _i = 0; _i < 2; ++_i) \
    __builtin_amdgcn_global_load_lds((const unsigned*)((const char*)(gbase) + voff + _i * 8192), (LAS unsigned*)(lds + (bufoff) + ldsw + _i * 8192), 16, 0, 0); } while (0)
#define LDA(dst, b, h) do { _Pragma("unroll") for (int m = 0; m < 4; ++m) _Pragma("unroll") for (int k = 0; k < 2; ++k) dst[m][k] = *(const LAS i32x4*)(lds + SA(b, h) + aoff + m * 2048 + k * 1024); } while (0)
#define LDB(dst, b, h) do { _Pragma("unroll") for (int n = 0; n < 2; ++n) _Pragma("unroll") for (int k = 0; k < 2; ++k) dst[n][k] = *(const LAS i32x4*)(lds + SB(b, h) + boff2 + n * 2048 + k * 1024); } while (0)
#define MMA(ai, bj, At, Bt) do { __builtin_amdgcn_s_setprio(1); _Pragma("unroll") for (int m = 0; m < 4; ++m) _Pragma("unroll") for (int n = 0; n < 2; ++n) _Pragma("unroll") for (int k = 0; k < 2; ++k) \
    acc[ai][bj][m][n] = __builtin_amdgcn_mfma_i32_16x16x64_i8(At[m][k], Bt[n][k], acc[ai][bj][m][n], 0, 0, 0); __builtin_amdgcn_s_setprio(0); } while (0)
#define WAIT_V(n) asm volatile("s_waitcnt vmcnt(" #n ")" ::: "memory")
#define WAIT_L(n) asm volatile("s_waitcnt lgkmcnt(" #n ")" ::: "memory")
#define BAR __builtin_amdgcn_s_barrier()
#define SCHED __builtin_amdgcn_sched_barrier(0)
    int ui = 0;
    Unit cur, nxt;
    if (!unit_at(0, cur)) return;
    nxt = cur;
    i32x4 acc[2][2][4][2];
#pragma unroll
    for (int a = 0; a < 2; ++a)
#pragma unroll
        for (int b = 0; b < 2; ++b)
#pragma unroll
            for (int m = 0; m < 4; ++m)
#pragma unroll
                for (int n = 0; n < 2; ++n) acc[a][b][m][n] = (i32x4){0, 0, 0, 0};
    i32x4 At[4][2], B0[2][2], B1[2][2];
    const char* cA = (const char*)stg + (size_t)cur.pm * KSTEP; const char* cB = (const char*)stg + (size_t)(NPX + cur.pn) * KSTEP;
    {
        const unsigned k0 = koff(0, rank), k1 = koff(1, rank);
        STAGE(SB(0, 0), cB + k0); STAGE(SA(0, 0), cA + k0); STAGE(SB(0, 1), cB + k0 + IMG); STAGE(SA(0, 1), cA + k0 + IMG);
        if (wr == 1) BAR;
        WAIT_V(4); BAR;
        STAGE(SB(1, 0), cB + k1); STAGE(SA(1, 0), cA + k1); STAGE(SB(1, 1), cB + k1 + IMG);
        WAIT_V(6); BAR;
    }
    for (;;) {
        const bool has_next = unit_at(ui + 1, nxt);
        const char* nA = (const char*)stg + (size_t)nxt.pm * KSTEP; const char* nB = (const char*)stg + (size_t)(NPX + nxt.pn) * KSTEP;
        DBG_STAMP(4 + 2 * ui);
        for (int u = 0; u < 32; u += 2) {
            const bool last = (u == 30);
            const int zn2 = last ? -1 : need_z(cur, u + 2), zn3 = last ? -1 : need_z(cur, u + 3);
            const char* a1 = cA + koff(u + 1, rank);
            const unsigned k2 = koff(last ? 0 : u + 2, rank), k3 = koff(last ? 1 : u + 3, rank);
            const char* a2 = (last ? nA : cA) + k2; const char* b2 = (last ? nB : cB) + k2;
            const char* a3 = (last ? nA : cA) + k3; const char* b3 = (last ? nB : cB) + k3;
            LDB(B0, 0, 0); SCHED; LDA(At, 0, 0); STAGE(SA(1, 1), a1 + IMG);
            WAIT_L(8); BAR; WAIT_L(0); MMA(0, 0, At, B0); BAR; SCHED;
            if (zn2 > zdone) gate(zn2);
            LDB(B1, 0, 1); STAGE(SB(0, 0), b2);
            BAR; WAIT_L(0); MMA(0, 1, At, B1); BAR;
            LDA(At, 0, 1); STAGE(SA(0, 0), a2);
            BAR; WAIT_L(0); MMA(1, 0, At, B0); BAR; SCHED;
            STAGE(SB(0, 1), b2 + IMG);
            WAIT_V(6); BAR; MMA(1, 1, At, B1); BAR;
            LDB(B0, 1, 0); SCHED; LDA(At, 1, 0); STAGE(SA(0, 1), a2 + IMG);
            WAIT_L(8); BAR; WAIT_L(0); MMA(0, 0, At, B0); BAR; SCHED;
            if (zn3 > zdone) gate(zn3);
            LDB(B1, 1, 1); STAGE(SB(1, 0), b3);
            BAR; WAIT_L(0); MMA(0, 1, At, B1); BAR;
            LDA(At, 1, 1); STAGE(SA(1, 0), a3);
            BAR; WAIT_L(0); MMA(1, 0, At, B0); BAR; SCHED;
            STAGE(SB(1, 1), b3 + IMG);
            WAIT_V(6); BAR; MMA(1, 1, At, B1); BAR;
        }
        if (wr == 0) BAR;
        DBG_STAMP(5 + 2 * ui);
        {
            const bool bad = *(volatile LAS unsigned*)(ctl + 1) != 0u;
            const float qn = __builtin_nanf("");
            unsigned ovo = (unsigned)(((wr * 64 + 4 * fq) * N + wc * 64 + 4 * fr) * 4); asm volatile("" : "+v"(ovo));
            const size_t toff = (size_t)cur.pm * 256 * N + cur.pn * 256;
            const __amdgpu_buffer_rsrc_t ors = __builtin_amdgcn_make_buffer_rsrc((void*)(P.out + toff), 0, 256u * N * 4u, 0x00020000);
#define EPILOGUE(AUX) do { _Pragma("unroll") for (int ai = 0; ai < 2; ++ai) _Pragma("unroll") for (int m = 0; m < 4; ++m) _Pragma("unroll") for (int r = 0; r < 4; ++r) { \
                f32x4 o = {(float)acc[ai][0][m][0][r] * scale, (float)acc[ai][0][m][1][r] * scale, (float)acc[ai][1][m][0][r] * scale, (float)acc[ai][1][m][1][r] * scale}; \
                if (bad) o = (f32x4){qn, qn, qn, qn}; \
                __builtin_amdgcn_raw_buffer_store_b128(__builtin_bit_cast(i32x4, o), ors, ovo + (unsigned)(ai * 128 + m * 16 + r) * N * 4u, 0, AUX); } } while (0)
            EPILOGUE(2);
            if (share) {
                WAIT_V(0); BAR;
                if (tid == 0) {
                    __builtin_amdgcn_fence(__ATOMIC_RELEASE, "agent");
                    asm volatile("s_waitcnt vmcnt(0)" ::: "memory");
                    __hip_atomic_fetch_add((GAS unsigned*)(ws + OFF_SHDONE), 1u, __ATOMIC_RELAXED, __HIP_MEMORY_SCOPE_AGENT);
                }
            }
        }
        if (!has_next) break;
#pragma unroll
        for (int a = 0; a < 2; ++a)
#pragma unroll
            for (int b = 0; b < 2; ++b)
#pragma unroll
                for (int m = 0; m < 4; ++m)
#pragma unroll
                    for (int n = 0; n < 2; ++n) acc[a][b][m][n] = (i32x4){0, 0, 0, 0};
        if (wr == 1) BAR;
        cur = nxt; cA = nA; cB = nB; ++ui;
    }
    WAIT_V(0);
    BAR;
    if (!share) { DBG_STAMP(3); return; }
    comm_phase(1);
}

extern "C" void kernel_launch(int rank, int nranks, ncclComm_t comm,
                              hipStream_t stream,
                              void* const* d_in, const int* in_sizes, int n_in,
                              void* d_out, int out_size,
                              const int* in_dtypes, int out_dtype,
                              void* const* d_out_all,
                              void* const* d_ws_all, size_t ws_bytes) {
    (void)comm; (void)in_sizes; (void)n_in; (void)out_size; (void)in_dtypes; (void)out_dtype;
    if (nranks != NR || ws_bytes < WS_NEED || (unsigned)rank >= (unsigned)NR) { fprintf(stderr, "unexpected configuration\n"); return; }
    static bool attr_set[NR] = {};
    if (!attr_set[rank]) {
        HIP_CHECK(hipFuncSetAttribute((const void*)fused_gather_gemm, hipFuncAttributeMaxDynamicSharedMemorySize, LDS_BYTES));
        attr_set[rank] = true;
    }
    Params p;
    p.x = (const signed char*)d_in[0]; p.w = (const signed char*)d_in[1];
    p.sx = (const float*)d_in[2]; p.sw = (const float*)d_in[3];
    p.out = (float*)d_out; p.ws_all = d_ws_all; p.out_all = d_out_all; p.rank = rank;
    hipLaunchKernelGGL(fused_gather_gemm, dim3(GRID), dim3(512), LDS_BYTES, stream, p);
    HIP_CHECK(hipGetLastError());
}
```
